```python
import numpy as np

kernel_name = "gpu_gemm_ar_m2048_k2048_n2048_bf16_none"


ABSMAX_RTOL = 0.008

COMPUTE_DTYPE = "bfloat16"
EPILOGUE = "none"
IN_QUANT = None
OUT_QUANT = None
IN_DTYPES = {"x": "bfloat16", "w_mat": "bfloat16"}
WORLD_SIZE = 8

def _to_bf16(x):
    u = np.ascontiguousarray(x, dtype=np.float32).view(np.uint32)
    return ((u + 0x7FFF + ((u >> 16) & 1)) & 0xFFFF0000).view(np.float32)

def setup_inputs():

    x = _to_bf16(np.random.randn(2048, 2048).astype(np.float32))
    w_mat = _to_bf16((np.random.randn(2048, 2048) / np.sqrt(2048)).astype(np.float32))
    return {"x": x, "w_mat": w_mat}

def _epilogue(y):
    return y

def reference(x, w_mat):
    y = (x @ w_mat).astype(np.float32)
    return _epilogue(y)
```

```cpp
#include <hip/hip_runtime.h>
#include <rccl/rccl.h>
#include <cstdio>
#include <cstdlib>

#define HIP_CHECK(expr) do { hipError_t _e = (expr); if (_e != hipSuccess) { \
    fprintf(stderr, "HIP error %s at %s:%d\n", hipGetErrorString(_e), __FILE__, __LINE__); \
    return; } } while (0)

namespace {

typedef unsigned u32x4 __attribute__((ext_vector_type(4)));
typedef unsigned u32x2 __attribute__((ext_vector_type(2)));
typedef float f32x4 __attribute__((ext_vector_type(4)));
typedef __bf16 bf16x8 __attribute__((ext_vector_type(8)));
typedef __bf16 bf16x2 __attribute__((ext_vector_type(2)));

constexpr int NR = 8;
constexpr int NCOMM = 64;
constexpr int NPIECE = 64;
constexpr unsigned IN_BYTES = 2048u * 256u * 2u;
constexpr unsigned OUT_BYTES = 2048u * 2048u * 2u;
constexpr unsigned XS_OFF = 0;
constexpr unsigned XS_BYTES = 4u << 20;
constexpr unsigned WS_OFF = XS_OFF + XS_BYTES;
constexpr unsigned WS_BYTES = 2u << 20;
constexpr unsigned FL_OFF = WS_OFF + WS_BYTES;
constexpr unsigned FX_OFF = FL_OFF;
constexpr unsigned FW_OFF = FX_OFF + 8 * 64 * 128;
constexpr unsigned FO_OFF = FW_OFF + 8 * 8 * 128;
constexpr unsigned CO_OFF = FO_OFF + 8 * 64 * 4;
constexpr unsigned SQ_OFF = CO_OFF + 64 * 128;
constexpr unsigned TMO_OFF = SQ_OFF + 256 * 64;
constexpr unsigned DBG_OFF = TMO_OFF + 128;
constexpr unsigned DBG_BYTES = 256 * 8 * 16 * 8;
constexpr unsigned WS_USED = DBG_OFF + DBG_BYTES;
constexpr unsigned long long DEADLINE_TICKS = 50000000ull;

constexpr int AUX_SYS = 17;

struct Params {
  char* ws[NR];
  char* out[NR];
  const char* x;
  const char* w;
  int rank;
  int nset;
  int win;
  int gate;
  int dmap;
};

#define RSRC(p, n) __builtin_amdgcn_make_buffer_rsrc((void*)(p), 0, (int)(n), 0x00020000)
#define VM0() asm volatile("s_waitcnt vmcnt(0)" ::: "memory")

#define STAMP(i) do { } while (0)

__device__ __forceinline__ unsigned ld_sys(const unsigned* p) {
  return __hip_atomic_load(p, __ATOMIC_RELAXED, __HIP_MEMORY_SCOPE_SYSTEM);
}
__device__ __forceinline__ void st_sys(unsigned* p, unsigned v) {
  __hip_atomic_store(p, v, __ATOMIC_RELAXED, __HIP_MEMORY_SCOPE_SYSTEM);
}

__device__ __forceinline__ bool wait_flag(const unsigned* f, unsigned seq, unsigned* tmo) {
  if (__builtin_amdgcn_readfirstlane(ld_sys(f)) >= seq) return true;
  const unsigned long long t0 = __builtin_amdgcn_s_memrealtime();
  for (unsigned it = 1;; ++it) {
    __builtin_amdgcn_s_sleep(2);
    if (__builtin_amdgcn_readfirstlane(ld_sys(f)) >= seq) return true;
    if (it & 63u) continue;
    const unsigned t = __hip_atomic_load(tmo, __ATOMIC_RELAXED, __HIP_MEMORY_SCOPE_AGENT);
    if (__builtin_amdgcn_readfirstlane(t) == seq || __builtin_amdgcn_s_memrealtime() - t0 > DEADLINE_TICKS) {
      __hip_atomic_store(tmo, seq, __ATOMIC_RELAXED, __HIP_MEMORY_SCOPE_AGENT);
      return false;
    }
  }
}

__global__ void __launch_bounds__(512, 2) gemm_ar_kernel(Params P) {
  __shared__ __attribute__((aligned(16))) float smem[4 * 8 * 1024 + 64];
  const int tid = threadIdx.x;
  const int lane = tid & 63;
  const int wave = __builtin_amdgcn_readfirstlane(tid >> 6);
  const int bid = blockIdx.x;
  const unsigned rank = (unsigned)P.rank & 7u;
  char* const myws = P.ws[0];
  unsigned* const sq = (unsigned*)(myws + SQ_OFF + bid * 64);
  unsigned* const tmo = (unsigned*)(myws + TMO_OFF);
  typedef __attribute__((address_space(3))) volatile unsigned lds_vu32;
  lds_vu32* const ctl = (lds_vu32*)(smem + 4 * 8 * 1024);
  STAMP(0);

  if (bid < NCOMM) {
    const unsigned dsel = P.dmap ? (unsigned)(bid >> 3) : (unsigned)(bid & 7);
    const unsigned q = P.dmap ? (unsigned)(bid & 7) : (unsigned)(bid >> 3);
    const unsigned d = (rank + dsel) & 7u;
    char* const dws = P.ws[dsel & 7u];
    const auto rx = RSRC(P.x, IN_BYTES);
    const auto rw = RSRC(P.w, IN_BYTES);
    const auto rd = RSRC(dws, FL_OFF);
    u32x4 a[8], b[8];
    const unsigned xrow0 = (d >> 2) * 1024u + (lane & 15);
    const unsigned xo = (lane >> 4) * 16u;
    if (wave < 4) {
      u32x2 t[16];
      const unsigned wbase = ((unsigned)(wave * 64) + (lane >> 4) * 8u) * 4096u + ((d & 3u) * 512u + q * 64u + (lane & 15) * 4u) * 2u;
#pragma unroll
      for (int i = 0; i < 16; ++i)
        t[i] = __builtin_amdgcn_raw_buffer_load_b64(rw, wbase + ((i >> 3) * 32 + (i & 7)) * 4096u, 0, 0);
      const unsigned u2 = (unsigned)(wave + 4) * 8u + q;
#pragma unroll
      for (int ks = 0; ks < 8; ++ks)
        b[ks] = __builtin_amdgcn_raw_buffer_load_b128(rx, (xrow0 + u2 * 16u) * 512u + ks * 64u + xo, 0, 0);
#pragma unroll
      for (int h = 0; h < 2; ++h) {
#pragma unroll
        for (int i = 0; i < 4; ++i) {
          const u32x2 lo = t[h * 8 + 2 * i], hi = t[h * 8 + 2 * i + 1];
          a[h * 4 + 0][i] = __builtin_amdgcn_perm(hi.x, lo.x, 0x05040100u);
          a[h * 4 + 1][i] = __builtin_amdgcn_perm(hi.x, lo.x, 0x07060302u);
          a[h * 4 + 2][i] = __builtin_amdgcn_perm(hi.y, lo.y, 0x05040100u);
          a[h * 4 + 3][i] = __builtin_amdgcn_perm(hi.y, lo.y, 0x07060302u);
        }
      }
    } else {
      const unsigned u1 = (unsigned)(wave - 4) * 8u + q;
#pragma unroll
      for (int ks = 0; ks < 8; ++ks)
        a[ks] = __builtin_amdgcn_raw_buffer_load_b128(rx, (xrow0 + u1 * 16u) * 512u + ks * 64u + xo, 0, 0);
    }
    unsigned seq = __builtin_amdgcn_readfirstlane(__hip_atomic_load(sq, __ATOMIC_RELAXED, __HIP_MEMORY_SCOPE_AGENT)) + 1u;
    asm volatile("" : "+s"(seq));
    if (tid < 16) ctl[tid] = 0u;
    __syncthreads();
    const int win = P.win;
    {
      const int k = wave;
      if (k >= win) { while (ctl[k - win] == 0u) __builtin_amdgcn_s_sleep(1); }
      STAMP(1);
      unsigned dst; unsigned* flag;
      if (wave < 4) {
        dst = WS_OFF + ((rank * 8u + q) * 32u + (unsigned)wave * 8u) * 1024u;
        flag = (unsigned*)(dws + FW_OFF + (rank * 8u + q) * 128u) + wave;
      } else {
        const unsigned u1 = (unsigned)(wave - 4) * 8u + q;
        dst = XS_OFF + (rank * 64u + u1) * 8192u;
        flag = (unsigned*)(dws + FX_OFF + (rank * 64u + u1) * 128u);
      }
#pragma unroll
      for (int i = 0; i < 8; ++i) __builtin_amdgcn_raw_buffer_store_b128(a[i], rd, dst + i * 1024u + lane * 16u, 0, AUX_SYS);
      VM0();
      if (lane == 0) { st_sys(flag, seq); ctl[k] = 1u; }
      STAMP(2);
    }
    if (wave < 4) {
      const int k = wave + 8;
      while (ctl[k - win] == 0u) __builtin_amdgcn_s_sleep(1);
      STAMP(3);
      const unsigned u2 = (unsigned)(wave + 4) * 8u + q;
      const unsigned dst = XS_OFF + (rank * 64u + u2) * 8192u;
      unsigned* flag = (unsigned*)(dws + FX_OFF + (rank * 64u + u2) * 128u);
#pragma unroll
      for (int i = 0; i < 8; ++i) __builtin_amdgcn_raw_buffer_store_b128(b[i], rd, dst + i * 1024u + lane * 16u, 0, AUX_SYS);
      VM0();
      if (lane == 0) { st_sys(flag, seq); ctl[k] = 1u; }
      STAMP(4);
    }
    if (d != rank) {
      if (P.gate > 0) { while (ctl[P.gate - 1] == 0u) __builtin_amdgcn_s_sleep(1); }
      STAMP(5);
      const auto rs = RSRC(P.out[dsel & 7u], OUT_BYTES);
      const auto ro = RSRC(P.out[0], OUT_BYTES);
      const unsigned* fo = (const unsigned*)(myws + FO_OFF) + d * 64u;
      const unsigned obase = ((d >> 2) * 1024u + q * 16u + (unsigned)wave * 2u) * 4096u + (d & 3u) * 1024u + lane * 16u;
      unsigned long long ready = 0ull;
      bool ok = true;
#pragma unroll 1
      for (int i = 0; i < 8; ++i) {
        const unsigned u = (unsigned)i * 8u + q;
        if (ok && !((ready >> u) & 1ull)) {
          ready = __ballot(ld_sys(fo + lane) >= seq);
          if (!((ready >> u) & 1ull)) { ok = wait_flag(fo + u, seq, tmo); }
        }
        if (i == 0) STAMP(6);
        const unsigned off = obase + (unsigned)i * (128u * 4096u);
        u32x4 v0 = {0x7fc07fc0u, 0x7fc07fc0u, 0x7fc07fc0u, 0x7fc07fc0u}, v1 = v0;
        if (ok) {
          v0 = __builtin_amdgcn_raw_buffer_load_b128(rs, off, 0, AUX_SYS);
          v1 = __builtin_amdgcn_raw_buffer_load_b128(rs, off + 4096u, 0, AUX_SYS);
        }
        __builtin_amdgcn_raw_buffer_store_b128(v0, ro, off, 0, 16);
        __builtin_amdgcn_raw_buffer_store_b128(v1, ro, off + 4096u, 0, 16);
      }
    }
    if (tid == 0) __hip_atomic_store(sq, seq, __ATOMIC_RELAXED, __HIP_MEMORY_SCOPE_AGENT);
    STAMP(7);
  } else {
    const unsigned s = (unsigned)bid & 7u;
    const unsigned set = (unsigned)(bid - NCOMM) >> 3;
    const unsigned p = (unsigned)wave;
    const auto rl = RSRC(myws, FL_OFF);
    const auto ro = RSRC(P.out[0], OUT_BYTES);
    unsigned seq = __builtin_amdgcn_readfirstlane(__hip_atomic_load(sq, __ATOMIC_RELAXED, __HIP_MEMORY_SCOPE_AGENT)) + 1u;
    asm volatile("" : "+s"(seq));
    if (tid == 0) ctl[0] = 0u;
    __syncthreads();
    if (tid == 0) __hip_atomic_store(sq, seq, __ATOMIC_RELAXED, __HIP_MEMORY_SCOPE_AGENT);
    char* const peer_ws = P.ws[lane & 7];
    bool ok = true;
    u32x4 B[32];
    {
      const unsigned* fw = (const unsigned*)(myws + FW_OFF + (p * 8u + s) * 128u);
      const unsigned wb = WS_OFF + (p * 8u + s) * 32768u + lane * 16u;
#pragma unroll
      for (int k = 0; k < 4; ++k) {
        if (ok) ok = wait_flag(fw + k, seq, tmo);
#pragma unroll
        for (int i = 0; i < 8; ++i) B[k * 8 + i] = __builtin_amdgcn_raw_buffer_load_b128(rl, wb + (k * 8 + i) * 1024u, 0, AUX_SYS);
      }
    }
    STAMP(1);
    int t = 0;
#pragma unroll 1
    for (unsigned u = set; u < (unsigned)NPIECE; u += (unsigned)P.nset, ++t) {
      if (ok) ok = wait_flag((const unsigned*)(myws + FX_OFF + (p * 64u + u) * 128u), seq, tmo);
      u32x4 A[8];
      const unsigned xb = XS_OFF + (p * 64u + u) * 8192u + lane * 16u;
#pragma unroll
      for (int ks = 0; ks < 8; ++ks) A[ks] = __builtin_amdgcn_raw_buffer_load_b128(rl, xb + ks * 1024u, 0, AUX_SYS);
      __builtin_amdgcn_sched_barrier(0);
      f32x4 acc[4] = {};
#pragma unroll
      for (int ks = 0; ks < 8; ++ks) {
#pragma unroll
        for (int c = 0; c < 4; ++c)
          acc[c] = __builtin_amdgcn_mfma_f32_16x16x32_bf16(__builtin_bit_cast(bf16x8, A[ks]), __builtin_bit_cast(bf16x8, B[ks * 4 + c]), acc[c], 0, 0, 0);
      }
      float* const buf = smem + (t & 3) * 8192;
      {
        float* const dst = buf + p * 1024 + (lane >> 4) * 256 + (lane & 15) * 4;
#pragma unroll
        for (int r = 0; r < 4; ++r) *(f32x4*)(dst + r * 64) = f32x4{acc[0][r], acc[1][r], acc[2][r], acc[3][r]};
      }
      if (!ok) ctl[0] = 1u;
      __syncthreads();
      if ((wave >> 1) == (t & 3)) {
        const int e = (wave & 1) * 64 + lane;
        const float* src = buf + e * 8;
        f32x4 s0 = *(const f32x4*)(src), s1 = *(const f32x4*)(src + 4);
#pragma unroll
        for (int j = 1; j < 8; ++j) { s0 += *(const f32x4*)(src + j * 1024); s1 += *(const f32x4*)(src + j * 1024 + 4); }
        bf16x8 o;
#pragma unroll
        for (int j = 0; j < 4; ++j) { o[j] = (__bf16)s0[j]; o[4 + j] = (__bf16)s1[j]; }
        u32x4 ov = __builtin_bit_cast(u32x4, o);
        if (ctl[0] != 0u) ov = u32x4{0x7fc07fc0u, 0x7fc07fc0u, 0x7fc07fc0u, 0x7fc07fc0u};
        const unsigned off = ((rank >> 2) * 1024u + u * 16u + (unsigned)(e >> 3)) * 4096u + (rank & 3u) * 1024u + s * 128u + (unsigned)(e & 7) * 16u;
        __builtin_amdgcn_raw_buffer_store_b128(ov, ro, off, 0, AUX_SYS);
        VM0();
        unsigned old = 0;
        if (lane == 0) old = __hip_atomic_fetch_add((unsigned*)(myws + CO_OFF + u * 128u), 1u, __ATOMIC_RELAXED, __HIP_MEMORY_SCOPE_AGENT);
        old = __builtin_amdgcn_readfirstlane(old);
        if (old + 1u == seq * 16u) {
          if (lane >= 1 && lane < NR) st_sys((unsigned*)(peer_ws + FO_OFF) + rank * 64u + u, seq);
        }
      }
    }
  }
}

}

extern "C" void kernel_launch(int rank, int nranks, ncclComm_t comm,
                              hipStream_t stream,
                              void* const* d_in, const int* in_sizes, int n_in,
                              void* d_out, int out_size,
                              const int* in_dtypes, int out_dtype,
                              void* const* d_out_all,
                              void* const* d_ws_all, size_t ws_bytes) {
  if (nranks != NR || (unsigned)rank >= (unsigned)NR || n_in < 2 || ws_bytes < WS_USED ||
      (size_t)out_size * 2 != OUT_BYTES || (size_t)in_sizes[0] * 2 != IN_BYTES || (size_t)in_sizes[1] * 2 != IN_BYTES) {
    fprintf(stderr, "kernel_launch: unexpected problem shape\n");
    return;
  }
  Params P;
  for (int i = 0; i < NR; ++i) { P.ws[i] = (char*)d_ws_all[(rank + i) & 7]; P.out[i] = (char*)d_out_all[(rank + i) & 7]; }
  P.x = (const char*)d_in[0];
  P.w = (const char*)d_in[1];
  P.rank = rank;
  P.nset = 16; P.win = 3; P.gate = 12; P.dmap = 0;
  gemm_ar_kernel<<<dim3(NCOMM + 8 * P.nset), dim3(512), 0, stream>>>(P);
  HIP_CHECK(hipGetLastError());
}
```

```cpp
#include <hip/hip_runtime.h>
#include <rccl/rccl.h>
#include <cstdio>
#include <cstdlib>

#define HIP_CHECK(expr) do { hipError_t _e = (expr); if (_e != hipSuccess) { \
    fprintf(stderr, "HIP error %s at %s:%d\n", hipGetErrorString(_e), __FILE__, __LINE__); \
    return; } } while (0)

namespace {

typedef unsigned u32x4 __attribute__((ext_vector_type(4)));
typedef unsigned u32x2 __attribute__((ext_vector_type(2)));
typedef float f32x4 __attribute__((ext_vector_type(4)));
typedef __bf16 bf16x8 __attribute__((ext_vector_type(8)));
typedef __attribute__((address_space(1))) unsigned gu32;

#define PWL 8
#define ORDER 1
#define PER 107
#define XF 64
constexpr int NR = 8;
constexpr int NWWG = 64;
constexpr int NSET = 16;
constexpr int NCWG = 8 * NSET;
constexpr int NPWG = 7 * PWL;
constexpr int NWG = NWWG + NCWG + NPWG;
constexpr int NPIECE = 64;
constexpr unsigned IN_BYTES = 2048u * 256u * 2u;
constexpr unsigned OUT_BYTES = 2048u * 2048u * 2u;
constexpr unsigned XS_OFF = 0;
constexpr unsigned WS_OFF = XS_OFF + (4u << 20);
constexpr unsigned XE_OFF = WS_OFF + (2u << 20);
constexpr unsigned WE_OFF = XE_OFF + (1u << 20);
constexpr unsigned OE_OFF = WE_OFF + (1u << 20);
constexpr unsigned HALF_BYTES = OE_OFF + (1u << 20);
constexpr unsigned TK_OFF = 2 * HALF_BYTES;
constexpr unsigned SQ_OFF = TK_OFF + 8 * 128;
constexpr unsigned TMO_OFF = SQ_OFF + 256 * 64;
constexpr unsigned DBG_OFF = TMO_OFF + 128;
constexpr unsigned DBG_BYTES = 256 * 8 * 64 * 4;
constexpr unsigned WS_USED = DBG_OFF + DBG_BYTES;
constexpr unsigned long long DEADLINE_TICKS = 50000000ull;
constexpr unsigned NANS = 0x7fc07fc0u;

constexpr int AUX_SYS = 17;
constexpr int AUX_DEV = 16;

struct Params {
  const char* x;
  const char* w;
  char* ws0;
  int rank;
  int pad;
  char* out;
  int p[6];
  char* ws[NR];
};

#define RSRC(p, n) __builtin_amdgcn_make_buffer_rsrc((void*)(p), 0, (int)(n), 0x00020000)
#define VM0() asm volatile("s_waitcnt vmcnt(0)" ::: "memory")

#define STAMP(i) do { } while (0)
#define FLUSH() do { } while (0)

__device__ __forceinline__ unsigned umin3(unsigned a, unsigned b, unsigned c) { return min(min(a, b), c); }
__device__ __forceinline__ unsigned min4(u32x4 v) { return min(umin3(v[0], v[1], v[2]), v[3]); }
__device__ __forceinline__ u32x4 nonzero(u32x4 v) {
#pragma unroll
  for (int i = 0; i < 4; ++i) v[i] = v[i] ? v[i] : 0x8000u;
  return v;
}

struct Spin {
  unsigned it = 0; unsigned long long t0 = 0;
  __device__ __forceinline__ bool fail(gu32* tmo, unsigned seq) {
    asm volatile("" ::: "memory");
    if (it == 0) t0 = __builtin_amdgcn_s_memrealtime();
    if ((++it & 63u) != 0u) return false;
    const unsigned t = __hip_atomic_load(tmo, __ATOMIC_RELAXED, __HIP_MEMORY_SCOPE_AGENT);
    if (__builtin_amdgcn_readfirstlane(t) == seq || __builtin_amdgcn_s_memrealtime() - t0 > DEADLINE_TICKS) {
      __hip_atomic_store(tmo, seq, __ATOMIC_RELAXED, __HIP_MEMORY_SCOPE_AGENT);
      return true;
    }
    return false;
  }
};

template <class R>
__device__ __forceinline__ bool load8(u32x4* F, R rs, unsigned off, int lane, bool ok, gu32* tmo, unsigned seq) {
  Spin sp;
  while (ok && __builtin_amdgcn_readfirstlane(__builtin_amdgcn_raw_buffer_load_b32(rs, off + 8188u, 0, AUX_SYS)) == 0u) {
    __builtin_amdgcn_s_sleep(2);
    if (sp.fail(tmo, seq)) ok = false;
  }
  for (;;) {
#pragma unroll
    for (int i = 0; i < 8; ++i) F[i] = __builtin_amdgcn_raw_buffer_load_b128(rs, off + (unsigned)lane * 16u + i * 1024u, 0, AUX_SYS);
    __builtin_amdgcn_sched_barrier(0);
    unsigned m = min4(F[0]);
#pragma unroll
    for (int i = 1; i < 8; ++i) m = min(m, min4(F[i]));
    if (!ok || __all(m != 0u)) return ok;
    __builtin_amdgcn_s_sleep(2);
    if (sp.fail(tmo, seq)) ok = false;
  }
}

__global__ void __launch_bounds__(512, 2) gemm_ar_kernel(Params P) {
  __shared__ __attribute__((aligned(16))) float smem[4 * 8 * 1024 + 64];
  const int tid = threadIdx.x;
  const int lane = tid & 63;
  STAMP(0);
  const int wave = __builtin_amdgcn_readfirstlane(tid >> 6);
  const int hb = blockIdx.x;
  const int bid = hb < 32 ? hb : hb < 32 + NPWG ? hb - 32 + NCWG + NWWG : hb < 32 + NPWG + NWWG ? hb - 32 - NPWG + NCWG : hb - NPWG - NWWG;
  const char* px = P.x; const char* pw = P.w; char* myws = P.ws0; unsigned rank = (unsigned)P.rank & 7u;
  char* outp = P.out; char* peer = P.ws[((unsigned)(bid - NWWG - NCWG) / PWL + 1u) & 7u];
  asm volatile("" : "+s"(px), "+s"(pw), "+s"(myws), "+s"(rank), "+s"(outp), "+s"(peer));
  gu32* const sq = (gu32*)(myws + SQ_OFF + bid * 64) + wave;
  gu32* const tmo = (gu32*)(myws + TMO_OFF);
  const unsigned seq_v = __hip_atomic_load(sq, __ATOMIC_RELAXED, __HIP_MEMORY_SCOPE_AGENT);
  const u32x4 nanv = {NANS, NANS, NANS, NANS};

  if (__builtin_expect(bid < NCWG, 1)) {
    typedef __attribute__((address_space(3))) volatile unsigned lds_vu32;
    lds_vu32* const ctl = (lds_vu32*)(smem + 4 * 8 * 1024);
    const unsigned cid = (unsigned)bid;
    const unsigned s = cid & 7u;
    const unsigned set = cid >> 3;
    const unsigned p = (unsigned)wave;
    const unsigned slot = (cid * 8u + (unsigned)wave) * 1024u + lane * 16u;
    const unsigned g = (cid & 1u) * 64u + (cid >> 1);
    const u32x4 xv = __builtin_amdgcn_raw_buffer_load_b128(RSRC(px, IN_BYTES), (g * 16u + (lane & 15)) * 512u + (unsigned)wave * 64u + (lane >> 4) * 16u, 0, 0);
    const unsigned seq = __builtin_amdgcn_readfirstlane(seq_v) + 1u;
    const auto rl = RSRC(myws + (seq & 1u) * HALF_BYTES, HALF_BYTES);
    const auto rz = RSRC(myws + ((seq & 1u) ^ 1u) * HALF_BYTES, HALF_BYTES);
    __builtin_amdgcn_raw_buffer_store_b128(nonzero(xv), rl, XE_OFF + (g * 8u + (unsigned)wave) * 1024u + lane * 16u, 0, AUX_SYS);
    STAMP(30);
    if (lane == 0) __hip_atomic_store(sq, seq, __ATOMIC_RELAXED, __HIP_MEMORY_SCOPE_AGENT);
    if (tid == 0) ctl[0] = 0u;
    __syncthreads();
    const u32x4 zero4 = {0u, 0u, 0u, 0u};
    __builtin_amdgcn_raw_buffer_store_b128(zero4, rz, XE_OFF + slot, 0, AUX_SYS);
    __builtin_amdgcn_raw_buffer_store_b128(zero4, rz, WE_OFF + slot, 0, AUX_SYS);
    bool ok = true;
    u32x4 B[32];
    {
      const unsigned wb = p == rank ? WE_OFF + ((rank & 3u) * 8u + s) * 32768u : WS_OFF + (p * 8u + s) * 32768u;
#pragma unroll
      for (int k = 0; k < 4; ++k) ok = load8(B + 8 * k, rl, wb + k * 8192u, lane, ok, tmo, seq);
    }
    STAMP(29);
    const unsigned xbase = p == rank ? XE_OFF + (rank >> 2) * 64u * 8192u : XS_OFF + p * 64u * 8192u;
    int t = 0;
#pragma unroll 1
    for (unsigned u = set; u < (unsigned)NPIECE; u += (unsigned)NSET, ++t) {
      u32x4 A[8];
      ok = load8(A, rl, xbase + u * 8192u, lane, ok, tmo, seq);
      STAMP(2 + t * 3);
      f32x4 acc[4] = {};
#pragma unroll
      for (int ks = 0; ks < 8; ++ks) {
#pragma unroll
        for (int c = 0; c < 4; ++c)
          acc[c] = __builtin_amdgcn_mfma_f32_16x16x32_bf16(__builtin_bit_cast(bf16x8, A[ks]), __builtin_bit_cast(bf16x8, B[ks * 4 + c]), acc[c], 0, 0, 0);
      }
      float* const buf = smem + (t & 3) * 8192;
      {
        float* const dst = buf + p * 1024 + (lane >> 4) * 256 + (lane & 15) * 4;
#pragma unroll
        for (int r = 0; r < 4; ++r) *(f32x4*)(dst + r * 64) = f32x4{acc[0][r], acc[1][r], acc[2][r], acc[3][r]};
      }
      if (!ok) ctl[0] = 1u;
      __syncthreads();
      STAMP(3 + t * 3);
      if (t == 0) __builtin_amdgcn_raw_buffer_store_b128(zero4, rz, OE_OFF + slot, 0, AUX_SYS);
      if ((wave >> 1) == (t & 3)) {
        const int e = (wave & 1) * 64 + lane;
        const float* src = buf + e * 8;
        f32x4 s0 = *(const f32x4*)(src), s1 = *(const f32x4*)(src + 4);
#pragma unroll
        for (int j = 1; j < 8; ++j) { s0 += *(const f32x4*)(src + j * 1024); s1 += *(const f32x4*)(src + j * 1024 + 4); }
        bf16x8 o;
#pragma unroll
        for (int j = 0; j < 4; ++j) { o[j] = (__bf16)s0[j]; o[4 + j] = (__bf16)s1[j]; }
        u32x4 ov = __builtin_bit_cast(u32x4, o);
        if (ctl[0] != 0u) ov = nanv;
        const unsigned row = u * 16u + (unsigned)(e >> 3), col = s * 128u + (unsigned)(e & 7) * 16u;
        __builtin_amdgcn_raw_buffer_store_b128(nonzero(ov), rl, OE_OFF + row * 1024u + col, 0, AUX_SYS);
        __builtin_amdgcn_raw_buffer_store_b128(ov, RSRC(outp, OUT_BYTES), ((rank >> 2) * 1024u + row) * 4096u + (rank & 3u) * 1024u + col, 0, AUX_DEV);
        STAMP(4 + t * 3);
      }
    }
#pragma unroll
    for (int i = 0; i < 6; ++i) __builtin_amdgcn_raw_buffer_store_b128(zero4, rz, XS_OFF + i * (1u << 20) + slot, 0, AUX_DEV);
    FLUSH();
  } else if (bid < NCWG + NWWG) {
    if (wave >= 4) return;
    const unsigned wid = (unsigned)(bid - NCWG);
    const unsigned ks = wid & 7u, q = (wid >> 3) * 4u + (unsigned)wave;
    const auto rw = RSRC(pw, IN_BYTES);
    u32x2 t[8];
    const unsigned wbase = (ks * 32u + (lane >> 4) * 8u) * 4096u + (q * 64u + (lane & 15) * 4u) * 2u;
#pragma unroll
    for (int i = 0; i < 8; ++i) t[i] = __builtin_amdgcn_raw_buffer_load_b64(rw, wbase + i * 4096u, 0, 0);
    __builtin_amdgcn_sched_barrier(0);
    const unsigned seq = __builtin_amdgcn_readfirstlane(seq_v) + 1u;
    const auto rl = RSRC(myws + (seq & 1u) * HALF_BYTES, HALF_BYTES);
    u32x4 a[4];
#pragma unroll
    for (int i = 0; i < 4; ++i) {
      const u32x2 lo = t[2 * i], hi = t[2 * i + 1];
      a[0][i] = __builtin_amdgcn_perm(hi.x, lo.x, 0x05040100u);
      a[1][i] = __builtin_amdgcn_perm(hi.x, lo.x, 0x07060302u);
      a[2][i] = __builtin_amdgcn_perm(hi.y, lo.y, 0x05040100u);
      a[3][i] = __builtin_amdgcn_perm(hi.y, lo.y, 0x07060302u);
    }
#pragma unroll
    for (int c = 0; c < 4; ++c) __builtin_amdgcn_raw_buffer_store_b128(nonzero(a[c]), rl, WE_OFF + ((q * 8u + ks) * 4u + c) * 1024u + lane * 16u, 0, AUX_SYS);
    STAMP(1);
    if (lane == 0) __hip_atomic_store(sq, seq, __ATOMIC_RELAXED, __HIP_MEMORY_SCOPE_AGENT);
    FLUSH();
  } else {
    const unsigned pid = (unsigned)(bid - NWWG - NCWG);
    const unsigned dsel = (pid / PWL + 1u) & 7u, q = pid % PWL;
    constexpr unsigned nwave = 8u;
    const unsigned d = (rank + dsel) & 7u;
    constexpr unsigned NWI = 256u, NSI = NWI + 512u, NITEM = NSI + 1024u;
    const char* wsrc = peer + WE_OFF + (rank & 3u) * (256u * 1024u);
    char* wdst = myws + WS_OFF + d * (256u * 1024u);
    const char* xsrc = peer + XE_OFF + (rank >> 2) * (512u * 1024u);
    char* xdst = myws + XS_OFF + d * (512u * 1024u);
    const char* osrc = peer + OE_OFF;
    char* odst = outp + (d >> 2) * (1024u * 4096u) + (d & 3u) * 1024u;
    unsigned* tkp = (unsigned*)(myws + TK_OFF + d * 128u);
    const unsigned one = 1u, zero = 0u;
#define TICKET() asm volatile("s_mov_b64 exec, 1\n\tglobal_atomic_add %0, %1, %2, %3 sc0\n\ts_mov_b64 exec, -1" : "=v"(tk) : "v"(zero), "v"(one), "s"(tkp) : "memory")
#define LOAD16() asm volatile("global_load_dwordx4 %0, %1, %2 sc0 sc1" : "=v"(v) : "v"(voff), "s"(src) : "memory")
    const char* src; char* dst;
    auto decode = [&](unsigned n) {
      if (n >= NSI) { src = osrc + (n - NSI) * 1024u; dst = odst + (n - NSI) * 4096u; }
      else if (n - XF < NWI) { src = wsrc + (n - XF) * 1024u; dst = wdst + (n - XF) * 1024u; }
      else { const unsigned m = n < XF ? n : n - NWI; src = xsrc + m * 1024u; dst = xdst + m * 1024u; }
    };
    decode((q * nwave + (unsigned)wave) & 127u);
    unsigned sv = seq_v;
    asm volatile("" : "+s"(wsrc), "+s"(wdst), "+s"(xsrc), "+s"(xdst), "+s"(osrc), "+s"(odst), "+s"(tkp), "+s"(src), "+s"(dst));
    asm volatile("" : "+v"(sv));
    const unsigned voff = (unsigned)lane * 16u + ((sv & 1u) ? 0u : HALF_BYTES);
    const unsigned phase = (q * nwave + (unsigned)wave) * PER / (PWL * nwave);
    auto slot_wait = [&]() {
      const unsigned now = (unsigned)__builtin_amdgcn_s_memrealtime();
      const unsigned left = PER - 1u - (now % PER + PER - phase - 1u) % PER;
      while ((unsigned)__builtin_amdgcn_s_memrealtime() - now < left) {}
    };
    slot_wait();
    STAMP(3);
    u32x4 v; unsigned tk;
    bool ok = true;
    LOAD16();
    TICKET();
    const unsigned seq = __builtin_amdgcn_readfirstlane(sv) + 1u;
    const unsigned tk_base = (seq - 1u) * NITEM - PWL * nwave;
    odst -= (seq & 1u) * HALF_BYTES;
    if (lane == 0) __hip_atomic_store(sq, seq, __ATOMIC_RELAXED, __HIP_MEMORY_SCOPE_AGENT);
#pragma unroll 1
    for (;;) {
      asm volatile("s_waitcnt vmcnt(0)" : "+v"(v), "+v"(tk) :: "memory");
      if (!__all(min4(v) != 0u)) {
        Spin sp;
        while (ok) {
          slot_wait();
          LOAD16();
          asm volatile("s_waitcnt vmcnt(0)" : "+v"(v) :: "memory");
          if (__all(min4(v) != 0u)) break;
          if (sp.fail(tmo, seq)) ok = false;
        }
        if (!ok) v = nanv;
      }
      const unsigned nn = __builtin_amdgcn_readfirstlane(tk) - tk_base;
      asm volatile("global_store_dwordx4 %0, %1, %2 sc1\n\ts_nop 1" :: "v"(voff), "v"(v), "s"(dst) : "memory");
      if (nn >= NITEM) break;
      decode(nn);
      LOAD16();
      TICKET();
    }
    STAMP(7);
    FLUSH();
  }
}

}

extern "C" void kernel_launch(int rank, int nranks, ncclComm_t comm,
                              hipStream_t stream,
                              void* const* d_in, const int* in_sizes, int n_in,
                              void* d_out, int out_size,
                              const int* in_dtypes, int out_dtype,
                              void* const* d_out_all,
                              void* const* d_ws_all, size_t ws_bytes) {
  if (nranks != NR || (unsigned)rank >= (unsigned)NR || n_in < 2 || ws_bytes < WS_USED ||
      (size_t)out_size * 2 != OUT_BYTES || (size_t)in_sizes[0] * 2 != IN_BYTES || (size_t)in_sizes[1] * 2 != IN_BYTES) {
    fprintf(stderr, "kernel_launch: unexpected problem shape\n");
    return;
  }
  Params P;
  for (int i = 0; i < NR; ++i) P.ws[i] = (char*)d_ws_all[(rank + i) & 7];
  P.out = (char*)d_out;
  P.x = (const char*)d_in[0];
  P.w = (const char*)d_in[1];
  P.rank = rank;
  P.ws0 = P.ws[0];
  P.pad = 0;
  for (int i = 0; i < 6; ++i) P.p[i] = 0;
  gemm_ar_kernel<<<dim3(NWG), dim3(512), 0, stream>>>(P);
  HIP_CHECK(hipGetLastError());
}
```

```cpp
#include <hip/hip_runtime.h>
#include <rccl/rccl.h>
#include <cstdio>
#include <cstdlib>

#define HIP_CHECK(expr) do { hipError_t _e = (expr); if (_e != hipSuccess) { \
    fprintf(stderr, "HIP error %s at %s:%d\n", hipGetErrorString(_e), __FILE__, __LINE__); \
    return; } } while (0)

namespace {

typedef unsigned u32x4 __attribute__((ext_vector_type(4)));
typedef unsigned u32x2 __attribute__((ext_vector_type(2)));
typedef float f32x4 __attribute__((ext_vector_type(4)));
typedef __bf16 bf16x8 __attribute__((ext_vector_type(8)));
typedef __attribute__((address_space(1))) unsigned gu32;

#define PWL 8
#define PER 107
#define STEAL 1
#define MINMV 12
#define ZSLEEP 16
#define RATE 145
#define XF 64
constexpr int NR = 8;
constexpr int NWWG = 64;
constexpr int NSET = 16;
constexpr int NCWG = 8 * NSET;
constexpr int NPWG = 7 * PWL;
constexpr int NWG = NWWG + NCWG + NPWG;
constexpr int NPIECE = 64;
constexpr unsigned IN_BYTES = 2048u * 256u * 2u;
constexpr unsigned OUT_BYTES = 2048u * 2048u * 2u;
constexpr unsigned XS_OFF = 0;
constexpr unsigned WS_OFF = XS_OFF + (4u << 20);
constexpr unsigned XE_OFF = WS_OFF + (2u << 20);
constexpr unsigned WE_OFF = XE_OFF + (1u << 20);
constexpr unsigned OE_OFF = WE_OFF + (1u << 20);
constexpr unsigned TK_OFF = OE_OFF + (1u << 20);
constexpr unsigned ZC_OFF = TK_OFF + 8 * 128;
constexpr unsigned PG_OFF = ZC_OFF;
constexpr unsigned HALF_BYTES = TK_OFF + 4096;
constexpr unsigned SQ_OFF = 2 * HALF_BYTES;
constexpr unsigned TMO_OFF = SQ_OFF + 256 * 64;
constexpr unsigned DBG_OFF = TMO_OFF + 128;
constexpr unsigned DBG_BYTES = 256 * 8 * 64 * 4;
constexpr unsigned WS_USED = DBG_OFF + DBG_BYTES;
constexpr unsigned long long DEADLINE_TICKS = 50000000ull;
constexpr unsigned NANS = 0x7fc07fc0u;
constexpr unsigned R0 = 128, RW = 96, RZ = 8 * RW;

constexpr int AUX_SYS = 17;
constexpr int AUX_DEV = 16;

struct Params {
  const char* x;
  const char* w;
  char* ws0;
  int rank;
  int pad;
  char* out;
  int p[6];
  char* ws[NR];
  char* outs[NR];
};

#define RSRC(p, n) __builtin_amdgcn_make_buffer_rsrc((void*)(p), 0, (int)(n), 0x00020000)
#define VM0() asm volatile("s_waitcnt vmcnt(0)" ::: "memory")

#define STAMP(i) do { } while (0)
#define FLUSH() do { } while (0)

__device__ __forceinline__ unsigned umin3(unsigned a, unsigned b, unsigned c) { return min(min(a, b), c); }
__device__ __forceinline__ unsigned min4(u32x4 v) { return min(umin3(v[0], v[1], v[2]), v[3]); }
__device__ __forceinline__ u32x4 nonzero(u32x4 v) {
#pragma unroll
  for (int i = 0; i < 4; ++i) v[i] = v[i] ? v[i] : 0x8000u;
  return v;
}

struct Spin {
  unsigned it = 0; unsigned long long t0 = 0;
  __device__ __forceinline__ bool fail(gu32* tmo, unsigned seq) {
    asm volatile("" ::: "memory");
    if (it == 0) t0 = __builtin_amdgcn_s_memrealtime();
    if ((++it & 63u) != 0u) return false;
    const unsigned t = __hip_atomic_load(tmo, __ATOMIC_RELAXED, __HIP_MEMORY_SCOPE_AGENT);
    if (__builtin_amdgcn_readfirstlane(t) == seq || __builtin_amdgcn_s_memrealtime() - t0 > DEADLINE_TICKS) {
      __hip_atomic_store(tmo, seq, __ATOMIC_RELAXED, __HIP_MEMORY_SCOPE_AGENT);
      return true;
    }
    return false;
  }
};

template <class R>
__device__ __forceinline__ bool load8(u32x4* F, R rs, unsigned off, int lane, bool ok, bool probe, gu32* tmo, unsigned seq) {
  Spin sp;
  for (;;) {
    while (probe && ok && __builtin_amdgcn_readfirstlane(__builtin_amdgcn_raw_buffer_load_b32(rs, off + 8188u, 0, AUX_SYS)) == 0u) {
      __builtin_amdgcn_s_sleep(2);
      if (sp.fail(tmo, seq)) ok = false;
    }
#pragma unroll
    for (int i = 0; i < 8; ++i) F[i] = __builtin_amdgcn_raw_buffer_load_b128(rs, off + (unsigned)lane * 16u + i * 1024u, 0, AUX_SYS);
    __builtin_amdgcn_sched_barrier(0);
    unsigned m = min4(F[0]);
#pragma unroll
    for (int i = 1; i < 8; ++i) m = min(m, min4(F[i]));
    if (!ok || __all(m != 0u)) return ok;
    probe = true;
    __builtin_amdgcn_s_sleep(2);
    if (sp.fail(tmo, seq)) ok = false;
  }
}

__global__ void __launch_bounds__(512, 2) gemm_ar_kernel(Params P) {
  __shared__ __attribute__((aligned(16))) float smem[4 * 8 * 1024 + 64];
  const int tid = threadIdx.x;
  const int lane = tid & 63;
  STAMP(0);
  const int wave = __builtin_amdgcn_readfirstlane(tid >> 6);
  const int bid = blockIdx.x;
  const char* px = P.x; const char* pw = P.w; char* myws = P.ws0; unsigned rank = (unsigned)P.rank & 7u; char* outp = P.out;
  const unsigned cid = (unsigned)(bid - NPWG);
  const unsigned g = (cid & 1u) * 64u + (cid >> 1);
  unsigned xoff = cid < (unsigned)NCWG ? (g * 16u + (lane & 15)) * 512u + (unsigned)wave * 64u + (lane >> 4) * 16u : IN_BYTES;
  unsigned sqoff = SQ_OFF + bid * 64 + wave * 4;
  asm volatile("" : "+v"(xoff), "+s"(sqoff));
  asm volatile("" : "+s"(px), "+s"(pw), "+s"(myws), "+s"(rank), "+s"(outp));
  const u32x4 xv = __builtin_amdgcn_raw_buffer_load_b128(RSRC(px, IN_BYTES), xoff, 0, 0);
  STAMP(50);
  gu32* const sq = (gu32*)(myws + sqoff);
  gu32* const tmo = (gu32*)(myws + TMO_OFF);
  const unsigned seq_v = __hip_atomic_load(sq, __ATOMIC_RELAXED, __HIP_MEMORY_SCOPE_AGENT);
  const u32x4 nanv = {NANS, NANS, NANS, NANS};

  __builtin_amdgcn_sched_barrier(0);
  if (cid < (unsigned)NCWG) {
    unsigned xe = XE_OFF + (g * 8u + (unsigned)wave) * 1024u + lane * 16u;
    asm volatile("" : "+v"(xe));
    __builtin_amdgcn_raw_buffer_store_b128(nonzero(xv), RSRC(myws, 2 * HALF_BYTES), xe + ((seq_v & 1u) ? 0u : HALF_BYTES), 0, AUX_SYS);
    STAMP(30);
  }
  unsigned role = (unsigned)bid;
  asm volatile("" : "+s"(role));
  if (role - NPWG < (unsigned)NCWG) {
    typedef __attribute__((address_space(3))) volatile unsigned lds_vu32;
    lds_vu32* const ctl = (lds_vu32*)(smem + 4 * 8 * 1024);
    const unsigned s = cid & 7u;
    const unsigned set = cid >> 3;
    const unsigned p = (unsigned)wave;
    const unsigned slot = (cid * 8u + (unsigned)wave) * 1024u + lane * 16u;
    const unsigned seq = __builtin_amdgcn_readfirstlane(seq_v) + 1u;
    const auto rl = RSRC(myws + (seq & 1u) * HALF_BYTES, HALF_BYTES);
    const auto rz = RSRC(myws + ((seq & 1u) ^ 1u) * HALF_BYTES, HALF_BYTES);
    if (lane == 0) __hip_atomic_store(sq, seq, __ATOMIC_RELAXED, __HIP_MEMORY_SCOPE_AGENT);
    if (tid == 0) ctl[0] = 0u;
    __syncthreads();
    const u32x4 zero4 = {0u, 0u, 0u, 0u};
    __builtin_amdgcn_raw_buffer_store_b128(zero4, rz, XE_OFF + slot, 0, AUX_SYS);
    __builtin_amdgcn_raw_buffer_store_b128(zero4, rz, WE_OFF + slot, 0, AUX_SYS);
    bool ok = true;
    u32x4 B[32];
    {
      const unsigned wb = p == rank ? WE_OFF + ((rank & 3u) * 8u + s) * 32768u : WS_OFF + (p * 8u + s) * 32768u;
#pragma unroll
      for (int k = 0; k < 4; ++k) ok = load8(B + 8 * k, rl, wb + k * 8192u, lane, ok, true, tmo, seq);
    }
    STAMP(29);
    const unsigned xbase = p == rank ? XE_OFF + (rank >> 2) * 64u * 8192u : XS_OFF + p * 64u * 8192u;
    int t = 0;
#pragma unroll 1
    for (unsigned u = set; u < (unsigned)NPIECE; u += (unsigned)NSET, ++t) {
      u32x4 A[8];
      unsigned zc = __builtin_amdgcn_raw_buffer_load_b32(rl, t == 3 && wave == 7 ? ZC_OFF : HALF_BYTES, 0, AUX_DEV);
      ok = load8(A, rl, xbase + u * 8192u, lane, ok, false, tmo, seq);
      STAMP(2 + t * 3);
      f32x4 acc[4] = {};
#pragma unroll
      for (int ks = 0; ks < 8; ++ks) {
#pragma unroll
        for (int c = 0; c < 4; ++c)
          acc[c] = __builtin_amdgcn_mfma_f32_16x16x32_bf16(__builtin_bit_cast(bf16x8, A[ks]), __builtin_bit_cast(bf16x8, B[ks * 4 + c]), acc[c], 0, 0, 0);
      }
      float* const buf = smem + (t & 3) * 8192;
      {
        float* const dst = buf + p * 1024 + (lane >> 4) * 256 + (lane & 15) * 4;
#pragma unroll
        for (int r = 0; r < 4; ++r) *(f32x4*)(dst + r * 64) = f32x4{acc[0][r], acc[1][r], acc[2][r], acc[3][r]};
      }
      if (STEAL && t == 3 && wave == 7) {
        Spin sp;
        while (ok && __builtin_amdgcn_readfirstlane(zc) < (unsigned)NWWG) {
          __builtin_amdgcn_s_sleep(32);
          if (sp.fail(tmo, seq)) ok = false;
          zc = __builtin_amdgcn_raw_buffer_load_b32(rl, ZC_OFF, 0, AUX_DEV);
        }
      }
      if (!ok) ctl[0] = 1u;
      __syncthreads();
      STAMP(3 + t * 3);
      if (t == 0) __builtin_amdgcn_raw_buffer_store_b128(zero4, rz, OE_OFF + slot, 0, AUX_SYS);
      if ((wave >> 1) == (t & 3)) {
        const int e = (wave & 1) * 64 + lane;
        const float* src = buf + e * 8;
        f32x4 s0 = *(const f32x4*)(src), s1 = *(const f32x4*)(src + 4);
#pragma unroll
        for (int j = 1; j < 8; ++j) { s0 += *(const f32x4*)(src + j * 1024); s1 += *(const f32x4*)(src + j * 1024 + 4); }
        bf16x8 o;
#pragma unroll
        for (int j = 0; j < 4; ++j) { o[j] = (__bf16)s0[j]; o[4 + j] = (__bf16)s1[j]; }
        u32x4 ov = __builtin_bit_cast(u32x4, o);
        if (ctl[0] != 0u) ov = nanv;
        const unsigned row = u * 16u + (unsigned)(e >> 3), col = s * 128u + (unsigned)(e & 7) * 16u;
        __builtin_amdgcn_raw_buffer_store_b128(nonzero(ov), rl, OE_OFF + row * 1024u + col, 0, AUX_SYS);
        __builtin_amdgcn_raw_buffer_store_b128(ov, RSRC(outp, OUT_BYTES), ((rank >> 2) * 1024u + row) * 4096u + (rank & 3u) * 1024u + col, 0, AUX_DEV);
        STAMP(4 + t * 3);
      }
    }
#pragma unroll
    for (int i = 0; i < 6; ++i) __builtin_amdgcn_raw_buffer_store_b128(zero4, rz, XS_OFF + i * (1u << 20) + slot, 0, AUX_DEV);
    FLUSH();
  } else if (role >= (unsigned)NPWG) {
    u32x2 wt[8];
    {
      const unsigned wi = (unsigned)(bid - NPWG - NCWG) & 63u;
      const unsigned wbase = wave < 4 ? ((wi & 7u) * 32u + (lane >> 4) * 8u) * 4096u + (((wi >> 3) * 4u + (unsigned)wave) * 64u + (lane & 15) * 4u) * 2u : IN_BYTES;
#pragma unroll
      for (int i = 0; i < 8; ++i) wt[i] = __builtin_amdgcn_raw_buffer_load_b64(RSRC(pw, IN_BYTES), wbase + i * 4096u, 0, 0);
      __builtin_amdgcn_sched_barrier(0);
    }
    const unsigned wid = (unsigned)(bid - NPWG - NCWG) & 63u;
    const unsigned seq = __builtin_amdgcn_readfirstlane(seq_v) + 1u;
    const auto rl = RSRC(myws + (seq & 1u) * HALF_BYTES, HALF_BYTES);
    if (wave < 4) {
      const unsigned ks = wid & 7u, q = (wid >> 3) * 4u + (unsigned)wave;
      u32x4 a[4];
#pragma unroll
      for (int i = 0; i < 4; ++i) {
        const u32x2 lo = wt[2 * i], hi = wt[2 * i + 1];
        a[0][i] = __builtin_amdgcn_perm(hi.x, lo.x, 0x05040100u);
        a[1][i] = __builtin_amdgcn_perm(hi.x, lo.x, 0x07060302u);
        a[2][i] = __builtin_amdgcn_perm(hi.y, lo.y, 0x05040100u);
        a[3][i] = __builtin_amdgcn_perm(hi.y, lo.y, 0x07060302u);
      }
#pragma unroll
      for (int c = 0; c < 4; ++c) __builtin_amdgcn_raw_buffer_store_b128(nonzero(a[c]), rl, WE_OFF + ((q * 8u + ks) * 4u + c) * 1024u + lane * 16u, 0, AUX_SYS);
    }
    STAMP(1);
    if (lane == 0) __hip_atomic_store(sq, seq, __ATOMIC_RELAXED, __HIP_MEMORY_SCOPE_AGENT);
    const u32x4 zero4 = {0u, 0u, 0u, 0u};
    if (wid == 0u && wave >= 6)
      __builtin_amdgcn_raw_buffer_store_b128(zero4, RSRC(myws + ((seq & 1u) ^ 1u) * HALF_BYTES, HALF_BYTES), TK_OFF + (unsigned)(wave - 6) * 1024u + lane * 16u, 0, AUX_DEV);
    if (STEAL) {
      __builtin_amdgcn_s_sleep(48);
      for (int i = wave; i > 0; --i) __builtin_amdgcn_s_sleep(ZSLEEP / 8);
      const auto ro = RSRC(outp, OUT_BYTES);
      const unsigned zi = wid * 8u + (unsigned)wave;
#pragma unroll 1
      for (unsigned i = 0; i < RZ * 8u / 512u; ++i) {
        const unsigned e = i * 512u + zi;
        const unsigned grp = e >= RZ * 4u ? 1u : 0u;
        if (grp * 4u + (e & 3u) != rank)
          __builtin_amdgcn_raw_buffer_store_b128(zero4, ro, (grp * (1024u - RZ) + R0) * 4096u + e * 1024u + lane * 16u, 0, AUX_SYS);
        __builtin_amdgcn_s_sleep(ZSLEEP);
      }
      VM0();
      __syncthreads();
      if (tid == 0) __hip_atomic_fetch_add((gu32*)(myws + (seq & 1u) * HALF_BYTES + ZC_OFF), 1u, __ATOMIC_RELAXED, __HIP_MEMORY_SCOPE_AGENT);
      STAMP(2);
    }
    FLUSH();
  } else {
    const unsigned pid = (unsigned)bid;
    const unsigned dsel = (pid / PWL + 1u) & 7u, q = pid % PWL;
    constexpr unsigned nwave = 8u;
    char* const peer = P.ws[dsel];
    const unsigned d = (rank + dsel) & 7u;
    constexpr unsigned NWI = 256u, NSI = NWI + 512u, NITEM = NSI + 1024u, NPW = PWL * nwave, MARK = 1280u;
    const char* wsrc = peer + WE_OFF + (rank & 3u) * (256u * 1024u);
    char* wdst = myws + WS_OFF + d * (256u * 1024u);
    const char* xsrc = peer + XE_OFF + (rank >> 2) * (512u * 1024u);
    char* xdst = myws + XS_OFF + d * (512u * 1024u);
    const char* osrc = peer + OE_OFF;
    char* tk0 = myws + TK_OFF;
    char* tkp = tk0 + dsel * 128u;
    const unsigned one = 1u;
#define FETCH() asm volatile("global_load_dwordx4 %0, %3, %4 sc0 sc1\n\tglobal_load_dword %2, %5, %6 offset:1024 sc1\n\t" \
      "s_mov_b64 exec, 1\n\tglobal_atomic_add %1, %3, %7, %8 sc0\n\ts_mov_b64 exec, -1" \
      : "=&v"(v), "=&v"(tk), "=&v"(cw) : "v"(voff), "s"(src), "v"(coff), "s"(cwp), "v"(one), "s"(tkp) : "memory")
#define LOAD16() asm volatile("global_load_dwordx4 %0, %1, %2 sc0 sc1" : "=v"(v) : "v"(voff), "s"(src) : "memory")
#define STORE16() asm volatile("global_store_dwordx4 %0, %1, %2 sc0 sc1\n\ts_nop 1" :: "v"(voff), "v"(v), "s"(dst) : "memory")
    auto rot = [&](unsigned j) { const unsigned w0 = R0 + RW * rank; return j < w0 ? j : j < 1024u - RW ? j + RW : j - (1024u - RW) + w0; };
    auto blk = [&](unsigned r) { return (r >> 2) * (1024u * 4096u) + (r & 3u) * 1024u; };
    char* obase = outp;
    const char* src; char* dst;
    auto decode = [&](unsigned n) {
      if (n >= NSI) { const unsigned row = rot(n - NSI); src = osrc + row * 1024u; dst = obase + blk(d) + row * 4096u; }
      else if (n - XF < NWI) {
        const unsigned m = n - XF, o = ((m >> 3) & 7u) * 32768u + (m >> 6) * 8192u + (m & 7u) * 1024u;
        src = wsrc + o; dst = wdst + o;
      } else { const unsigned m = n < XF ? n : n - NWI; src = xsrc + m * 1024u; dst = xdst + m * 1024u; }
    };
    unsigned n = (q * nwave + (unsigned)wave) & 127u;
    decode(n);
    unsigned sv = seq_v;
    asm volatile("" : "+s"(wsrc), "+s"(wdst), "+s"(xsrc), "+s"(xdst), "+s"(osrc), "+s"(tk0), "+s"(tkp), "+s"(src), "+s"(dst));
    asm volatile("" : "+v"(sv));
    const unsigned voff = (unsigned)lane * 16u + ((sv & 1u) ? 0u : HALF_BYTES);
    const unsigned coff = (unsigned)(lane & 7) * 4u + ((sv & 1u) ? 0u : HALF_BYTES);
    const unsigned phase = n * PER / NPW;
    auto slot_wait = [&]() {
      const unsigned now = (unsigned)__builtin_amdgcn_s_memrealtime();
      const unsigned left = PER - 1u - (now % PER + PER - phase - 1u) % PER;
      while ((unsigned)__builtin_amdgcn_s_memrealtime() - now < left) {}
    };
    slot_wait();
    STAMP(3);
    u32x4 v; unsigned tk, cw;
    char* cwp = tk0 + 128;
    bool ok = true;
    FETCH();
    const unsigned seq = __builtin_amdgcn_readfirstlane(sv) + 1u;
    const unsigned hoff = (seq & 1u) * HALF_BYTES;
    obase -= hoff;
    gu32* const ctw = (gu32*)(myws + hoff);
    if (lane == 0) __hip_atomic_store(sq, seq, __ATOMIC_RELAXED, __HIP_MEMORY_SCOPE_AGENT);
    auto settle = [&]() {
      Spin sp;
      while (ok) {
        slot_wait();
        LOAD16();
        asm volatile("s_waitcnt vmcnt(0)" : "+v"(v) :: "memory");
        if (__all(min4(v) != 0u)) break;
        if (sp.fail(tmo, seq)) ok = false;
      }
      if (!ok) v = nanv;
    };
    bool cleared = !STEAL, planned = !STEAL;
    unsigned lim = NITEM, lo = 0u;
    unsigned gp = 0u, gn = 0u;
    const char* direct = src;
    const char* const pout = P.outs[dsel] - hoff;
    auto plan = [&](unsigned pg) {
      const bool on = lane >= 1 && lane < NR;
      const int rel = on ? (int)(pg - __builtin_amdgcn_readlane(pg, 1)) : 0;
      const int a = max(min(rel, 4000), -4000) * RATE;
      int sum = 0;
#pragma unroll
      for (int l = 1; l < NR; ++l) sum += __builtin_amdgcn_readlane(a, l);
      const int ex = on ? (a - sum / 7) >> 8 : 0;
      const unsigned g = ex >= MINMV ? min((unsigned)ex, RW) : 0u, c = ex < 0 ? (unsigned)-ex : 0u;
      unsigned G = 0u, C = 0u, cp = 0u;
#pragma unroll
      for (int l = 1; l < NR; ++l) {
        gp = lane == l ? G : gp;
        if ((unsigned)l == dsel) cp = C;
        G += __builtin_amdgcn_readlane(g, l); C += __builtin_amdgcn_readlane(c, l);
      }
      if (G != 0u && C != 0u) {
        gn = g;
        lo = G * cp / C;
        lim = NITEM - __builtin_amdgcn_readlane(g, dsel) + (G * (cp + __builtin_amdgcn_readlane(c, dsel)) / C - lo);
      }
      planned = true;
    };
    bool isout = false;
    bool trusted = false;
#pragma unroll 1
    for (;;) {
      unsigned pg;
      asm volatile("s_waitcnt vmcnt(0)\n\tv_mov_b32 %2, %3" : "+v"(v), "+v"(tk), "=&v"(pg) : "v"(cw) : "memory");
      if (!__all(min4(v) != 0u)) {
        src = direct;
        settle();
      }
      const unsigned nn = __builtin_amdgcn_readfirstlane(tk) + NPW;
      if (!cleared && isout) {
        Spin sp;
        while (ok && __builtin_amdgcn_readfirstlane(pg) < (unsigned)NWWG) {
          __builtin_amdgcn_s_sleep(32);
          if (sp.fail(tmo, seq)) ok = false;
          asm volatile("global_load_dword %0, %1, %2 offset:1024 sc1\n\ts_waitcnt vmcnt(0)" : "=&v"(pg) : "v"(coff), "s"(tk0) : "memory");
        }
        if (!ok) v = nanv;
        cleared = true;
      }
      STORE16();
      if (!planned && nn >= NITEM - RW) {
        Spin sp;
        while (ok && !__all(pg != 0u)) {
          __builtin_amdgcn_s_sleep(16);
          if (sp.fail(tmo, seq)) ok = false;
          asm volatile("global_load_dword %0, %1, %2 offset:1024 sc1\n\ts_waitcnt vmcnt(0)" : "=&v"(pg) : "v"(coff), "s"(tk0) : "memory");
        }
        if (ok) plan(pg);
        planned = true;
      }
      if (nn >= lim) break;
      trusted = trusted || (ok && n >= NSI && rot(n - NSI) >= 768u);
      if (nn < NITEM) {
        decode(nn);
        direct = src;
      } else {
        const unsigned e = lo + (nn - NITEM);
        const unsigned vic = (unsigned)__builtin_ctzll(__ballot(e - gp < gn) | (1ull << 7));
        const unsigned row = rot(1024u - __builtin_amdgcn_readlane(gn, vic) + (e - __builtin_amdgcn_readlane(gp, vic))) & 1023u;
        const unsigned off = blk((rank + vic) & 7u) + row * 4096u;
        direct = P.ws[vic & 7u] + OE_OFF + row * 1024u;
        src = trusted && ok ? pout + off : direct;
        dst = obase + off;
      }
      cwp = (!cleared && nn >= NSI + R0) || (!planned && nn >= NITEM - RW - 4u * NPW) ? tk0 : tk0 + 128;
      FETCH();
      n = nn < NITEM ? nn : n;
      isout = nn >= NSI + R0;
      if (STEAL && nn >= MARK) {
        if (nn == MARK) __hip_atomic_store(ctw + PG_OFF / 4 + dsel, (unsigned)__builtin_amdgcn_s_memrealtime() | 1u, __ATOMIC_RELAXED, __HIP_MEMORY_SCOPE_AGENT);
        if (!planned && __all(pg != 0u)) plan(pg);
      }
    }
    STAMP(7);
    FLUSH();
  }
}

}

extern "C" void kernel_launch(int rank, int nranks, ncclComm_t comm,
                              hipStream_t stream,
                              void* const* d_in, const int* in_sizes, int n_in,
                              void* d_out, int out_size,
                              const int* in_dtypes, int out_dtype,
                              void* const* d_out_all,
                              void* const* d_ws_all, size_t ws_bytes) {
  if (nranks != NR || (unsigned)rank >= (unsigned)NR || n_in < 2 || ws_bytes < WS_USED ||
      (size_t)out_size * 2 != OUT_BYTES || (size_t)in_sizes[0] * 2 != IN_BYTES || (size_t)in_sizes[1] * 2 != IN_BYTES) {
    fprintf(stderr, "kernel_launch: unexpected problem shape\n");
    return;
  }
  Params P;
  for (int i = 0; i < NR; ++i) P.ws[i] = (char*)d_ws_all[(rank + i) & 7];
  P.out = (char*)d_out;
  P.x = (const char*)d_in[0];
  P.w = (const char*)d_in[1];
  P.rank = rank;
  P.ws0 = P.ws[0];
  for (int i = 0; i < NR; ++i) P.outs[i] = (char*)d_out_all[(rank + i) & 7];
  P.pad = 0;
  for (int i = 0; i < 6; ++i) P.p[i] = 0;
  gemm_ar_kernel<<<dim3(NWG), dim3(512), 0, stream>>>(P);
  HIP_CHECK(hipGetLastError());
}
```

```cpp
#include <hip/hip_runtime.h>
#include <rccl/rccl.h>
#include <cstdio>
#include <cstdlib>

#define HIP_CHECK(expr) do { hipError_t _e = (expr); if (_e != hipSuccess) { \
    fprintf(stderr, "HIP error %s at %s:%d\n", hipGetErrorString(_e), __FILE__, __LINE__); \
    return; } } while (0)

namespace {

typedef unsigned u32x4 __attribute__((ext_vector_type(4)));
typedef unsigned u32x2 __attribute__((ext_vector_type(2)));
typedef float f32x4 __attribute__((ext_vector_type(4)));
typedef __bf16 bf16x8 __attribute__((ext_vector_type(8)));
typedef __attribute__((address_space(1))) unsigned gu32;

#define PER 107
#define ZSLEEP 12
#define RATE 153
#define BPROBE 8188u
#define QUICK 2u
#define QUICKP 2u
#define NPUSH 64u
#define FINE 0
#define PBIAS 24
constexpr int NR = 8;
constexpr int PWL = 8;
constexpr int NPWG = 7 * PWL;
constexpr int NSET = 16;
constexpr int NCWG = 8 * NSET;
constexpr int NWWG = 64;
constexpr int NWG = NWWG + NCWG + NPWG;
constexpr unsigned IN_BYTES = 2048u * 256u * 2u;
constexpr unsigned OUT_BYTES = 2048u * 2048u * 2u;
constexpr unsigned XS_OFF = 0;
constexpr unsigned WS_OFF = XS_OFF + (2u << 20);
constexpr unsigned FE_OFF = WS_OFF + (2u << 20);
constexpr unsigned XE_OFF = FE_OFF + (2u << 20);
constexpr unsigned WE_OFF = XE_OFF + (1u << 20);
constexpr unsigned PE_OFF = WE_OFF + (1u << 20);
constexpr unsigned OE_OFF = PE_OFF + (1u << 20);
constexpr unsigned TK_OFF = OE_OFF + (1u << 20);
constexpr unsigned CW_OFF = TK_OFF + 8 * 128;
constexpr unsigned HALF_BYTES = TK_OFF + 4096;
constexpr unsigned SQ_OFF = 2 * HALF_BYTES;
constexpr unsigned TMO_OFF = SQ_OFF + 256 * 64;
constexpr unsigned DBG_OFF = TMO_OFF + 128;
constexpr unsigned DBG_BYTES = 256 * 8 * 64 * 4;
constexpr unsigned WS_USED = DBG_OFF + DBG_BYTES;
constexpr unsigned long long DEADLINE_TICKS = 50000000ull;
constexpr unsigned NANS = 0x7fc07fc0u;
constexpr unsigned long long REG_TC = 0xFEDCB73A98625410ull;
constexpr unsigned long long REG_OF = 0xFEDCB876A5329410ull;
constexpr unsigned CAP = 832;

constexpr int AUX_SYS = 17;
constexpr int AUX_DEV = 16;

struct Params {
  const char* x;
  const char* w;
  char* ws0;
  int rank;
  int pad;
  char* out;
  int p[6];
  char* ws[NR];
  char* outs[NR];
};

#define RSRC(p, n) __builtin_amdgcn_make_buffer_rsrc((void*)(p), 0, (int)(n), 0x00020000)
#define VM0() asm volatile("s_waitcnt vmcnt(0)" ::: "memory")

#define STAMP(i) do { } while (0)
#define FLUSH() do { } while (0)

__device__ __forceinline__ unsigned min4(u32x4 v) { return min(min(min(v[0], v[1]), v[2]), v[3]); }
__device__ __forceinline__ u32x4 nonzero(u32x4 v) {
#pragma unroll
  for (int i = 0; i < 4; ++i) v[i] = v[i] ? v[i] : 0x80000000u;
  return v;
}
__device__ __forceinline__ u32x4 to_bf16x8(f32x4 a, f32x4 b) {
  bf16x8 o;
#pragma unroll
  for (int j = 0; j < 4; ++j) { o[j] = (__bf16)a[j]; o[4 + j] = (__bf16)b[j]; }
  return __builtin_bit_cast(u32x4, o);
}
__device__ __forceinline__ unsigned out_off(unsigned o, unsigned f) {
  const unsigned tc = (unsigned)(REG_TC >> ((f >> 6) * 4u)) & 15u;
  const unsigned row = (o >> 2) * 1024u + (tc >> 2) * 256u + ((f >> 3) & 7u) * 32u + (o & 1u) * 16u + (f & 3u) * 4u;
  return row * 4096u + ((o >> 1) & 1u) * 2048u + (tc & 3u) * 512u + ((f >> 2) & 1u) * 256u;
}

struct Spin {
  unsigned it = 0; unsigned long long t0 = 0;
  __device__ __forceinline__ bool fail(gu32* tmo, unsigned seq) {
    asm volatile("" ::: "memory");
    if (it == 0) t0 = __builtin_amdgcn_s_memrealtime();
    if ((++it & 63u) != 0u) return false;
    const unsigned t = __hip_atomic_load(tmo, __ATOMIC_RELAXED, __HIP_MEMORY_SCOPE_AGENT);
    if (__builtin_amdgcn_readfirstlane(t) == seq || __builtin_amdgcn_s_memrealtime() - t0 > DEADLINE_TICKS) {
      __hip_atomic_store(tmo, seq, __ATOMIC_RELAXED, __HIP_MEMORY_SCOPE_AGENT);
      return true;
    }
    return false;
  }
};

template <int N, class R>
__device__ __forceinline__ bool load_frags(u32x4* F, R rs, unsigned off, int lane, bool ok, int probe, gu32* tmo, unsigned seq) {
  Spin sp;
  for (;;) {
    while (probe && ok) {
      const unsigned w = __builtin_amdgcn_raw_buffer_load_b32(rs, (probe == 1 ? (unsigned)(lane & (N - 1)) * 1024u : 0u) + 1020u, off, AUX_SYS);
      if (__all(w != 0u)) { --probe; continue; }
      __builtin_amdgcn_s_sleep(2);
      if (sp.fail(tmo, seq)) ok = false;
    }
#pragma unroll
    for (int i = 0; i < N; ++i) F[i] = __builtin_amdgcn_raw_buffer_load_b128(rs, (unsigned)lane * 16u + (i & 3) * 1024u, off + (i >> 2) * 4096u, AUX_SYS);
    __builtin_amdgcn_sched_barrier(0);
    unsigned m = min4(F[0]);
#pragma unroll
    for (int i = 1; i < N; ++i) m = min(m, min4(F[i]));
    if (!ok || __all(m != 0u)) return ok;
    probe = 1;
    if (sp.fail(tmo, seq)) ok = false;
  }
}

__global__ void __launch_bounds__(512, 2) gemm_ar_kernel(Params P) {
  __shared__ __attribute__((aligned(16))) float smem[2 * 8 * 2048 + 64];
  const int tid = threadIdx.x;
  const int lane = tid & 63;
  STAMP(0);
  const int wave = __builtin_amdgcn_readfirstlane(tid >> 6);
  const int bid = blockIdx.x;
  const char* px = P.x; const char* pw = P.w; char* myws = P.ws0; unsigned rank = (unsigned)P.rank & 7u; char* outp = P.out;
  const unsigned cid = (unsigned)(bid - NPWG);
  const unsigned g = (cid & 1u) * 64u + (cid & 6u) * 8u + (cid >> 3);
  unsigned xoff = cid < (unsigned)NCWG ? (g * 16u + (lane & 15)) * 512u + (unsigned)wave * 64u + (lane >> 4) * 16u : IN_BYTES;
  unsigned sqoff = SQ_OFF + bid * 64 + wave * 4;
  asm volatile("" : "+v"(xoff), "+s"(sqoff));
  asm volatile("" : "+s"(px), "+s"(pw), "+s"(myws), "+s"(rank), "+s"(outp));
  const u32x4 xv = __builtin_amdgcn_raw_buffer_load_b128(RSRC(px, IN_BYTES), xoff, 0, 0);
  gu32* const sq = (gu32*)(myws + sqoff);
  gu32* const tmo = (gu32*)(myws + TMO_OFF);
  const unsigned seq_v = __hip_atomic_load(sq, __ATOMIC_RELAXED, __HIP_MEMORY_SCOPE_AGENT);
  const u32x4 nanv = {NANS, NANS, NANS, NANS};
  const u32x4 zero4 = {0u, 0u, 0u, 0u};
  __builtin_amdgcn_sched_barrier(0);
  if (cid < (unsigned)NCWG) {
    unsigned xe = XE_OFF + (((unsigned)wave >> 2) * 512u + g * 4u + ((unsigned)wave & 3u)) * 1024u + lane * 16u;
    asm volatile("" : "+v"(xe));
    __builtin_amdgcn_raw_buffer_store_b128(nonzero(xv), RSRC(myws, 2 * HALF_BYTES), xe + ((seq_v & 1u) ? 0u : HALF_BYTES), 0, AUX_SYS);
    STAMP(30);
  }
  unsigned role = (unsigned)bid;
  asm volatile("" : "+s"(role));

  if (role - NPWG < (unsigned)NCWG) {
    typedef __attribute__((address_space(3))) volatile unsigned lds_vu32;
    lds_vu32* const ctl = (lds_vu32*)(smem + 2 * 8 * 2048);
    const unsigned s = cid & 7u, set = cid >> 3, v = (unsigned)wave;
    const unsigned slot = (cid * 8u + v) * 1024u;
    const unsigned seq = __builtin_amdgcn_readfirstlane(seq_v) + 1u;
    const auto rl = RSRC(myws + (seq & 1u) * HALF_BYTES, HALF_BYTES);
    const auto rz = RSRC(myws + ((seq & 1u) ^ 1u) * HALF_BYTES, HALF_BYTES);
    if (lane == 0) __hip_atomic_store(sq, seq, __ATOMIC_RELAXED, __HIP_MEMORY_SCOPE_AGENT);
    const auto rp = RSRC(P.ws[1] + (seq & 1u) * HALF_BYTES, HALF_BYTES);
    if (tid == 0) ctl[0] = 0u;
    bool ok = true;
    const unsigned xbase = (v < 2u ? XE_OFF + (v * 2u + (rank >> 2)) * (256u * 1024u) : XS_OFF + v * (256u * 1024u)) + set * 4096u;
    const unsigned wbase = (v < 2u ? WE_OFF + (v * 2u + ((rank >> 1) & 1u)) * (256u * 1024u) : WS_OFF + v * (256u * 1024u)) + ((s & 1u) * 4u + (s >> 1)) * 16384u;
    const bool own = (set & 1u) == (rank & 1u);
    u32x4 A0[4] = {}, A1[4], B0[16] = {}, B1[16];
    auto prefetch4 = [&](u32x4* F, unsigned off) {
#pragma unroll
      for (int i = 0; i < 4; ++i) F[i] = __builtin_amdgcn_raw_buffer_load_b128(rl, (unsigned)lane * 16u + i * 1024u, off, AUX_SYS);
    };
    auto full4 = [&](const u32x4* F) { return __all(min(min(min4(F[0]), min4(F[1])), min(min4(F[2]), min4(F[3]))) != 0u); };
    auto settle4 = [&](u32x4* F, unsigned off) {
      if (!full4(F)) ok = load_frags<4>(F, rl, off, lane, ok, 1, tmo, seq);
    };
    auto mul = [&](const u32x4* A, const u32x4* B, float* dst) {
      f32x4 acc[4] = {};
#pragma unroll
      for (int ks = 0; ks < 4; ++ks) {
#pragma unroll
        for (int c = 0; c < 4; ++c)
          acc[c] = __builtin_amdgcn_mfma_f32_16x16x32_bf16(__builtin_bit_cast(bf16x8, A[ks]), __builtin_bit_cast(bf16x8, B[ks * 4 + c]), acc[c], 0, 0, 0);
      }
#pragma unroll
      for (int r = 0; r < 4; ++r) *(f32x4*)(dst + r * 64) = f32x4{acc[0][r], acc[1][r], acc[2][r], acc[3][r]};
    };
    auto finish = [&](unsigned pb, unsigned nt, unsigned t0, unsigned cg0, unsigned t1, unsigned cg1, unsigned drop) {
      if (!ok) ctl[0] = 1u;
      __syncthreads();
      if ((v >> 2) == pb && ((v >> 1) & 1u) < nt) {
        const unsigned hi = (v >> 1) & 1u, e = (v & 1u) * 64u + (unsigned)lane;
        const float* src = smem + pb * 16384u + hi * 1024u + e * 8u;
        f32x4 r0[8], r1[8];
        const bool bad = ctl[0] != 0u;
#pragma unroll
        for (int j = 0; j < 8; ++j) { r0[j] = *(const f32x4*)(src + j * 2048); r1[j] = *(const f32x4*)(src + j * 2048 + 4); }
        __builtin_amdgcn_sched_barrier(0);
        const f32x4 s0 = ((r0[0] + r0[1]) + (r0[2] + r0[3])) + ((r0[4] + r0[5]) + (r0[6] + r0[7]));
        const f32x4 s1 = ((r1[0] + r1[1]) + (r1[2] + r1[3])) + ((r1[4] + r1[5]) + (r1[6] + r1[7]));
        const unsigned reg = (unsigned)(REG_OF >> ((hi ? t1 * 4u + cg1 : t0 * 4u + cg0) * 4u)) & 15u;
        const unsigned f = reg * 64u + (FINE && reg == 1u ? (s >> 2) * 32u + (set >> 1) * 4u : (set >> 1) * 8u + (s >> 2) * 4u) + (e >> 5);
        const unsigned il = (((e >> 3) & 3u) * 16u + ((s >> 1) & 1u) * 8u + (e & 7u)) * 16u;
        u32x4 o0 = bad ? nanv : nonzero(own ? __builtin_bit_cast(u32x4, s0) : to_bf16x8(s0, s1));
        u32x4 o1 = bad ? nanv : nonzero(__builtin_bit_cast(u32x4, s1));
        asm volatile("" : "+v"(o0), "+v"(o1));
        if (!drop) {
          __builtin_amdgcn_raw_buffer_store_b128(o0, !own && reg * 64u < NPUSH ? rp : rl, (own ? FE_OFF + f * 2048u : PE_OFF + f * 1024u) + il, 0, AUX_SYS);
          if (own) __builtin_amdgcn_raw_buffer_store_b128(o1, rl, FE_OFF + f * 2048u + 1024u + il, 0, AUX_SYS);
        }
      }
    };
    auto clear = [&](unsigned off) { __builtin_amdgcn_raw_buffer_store_b128(zero4, rz, (unsigned)lane * 16u, off + slot, AUX_SYS); };
    float* const l0 = smem + v * 2048u + (lane >> 4) * 256 + (lane & 15) * 4;
    float* const l1 = l0 + 16384;
    const unsigned g0 = s & 1u, g1 = g0 + 2u;
#pragma unroll 1
    for (unsigned it = 0; it < 2u; ++it) {
      if (it) {
        __syncthreads();
        Spin sp;
        while (ok && !__all(__builtin_amdgcn_raw_buffer_load_b32(rl, BPROBE, wbase, AUX_SYS) != 0u)) {
          __builtin_amdgcn_s_sleep(2);
          if (sp.fail(tmo, seq)) ok = false;
        }
        STAMP(27);
        unsigned todo = 31u;
        for (;;) {
          if (todo & 16u) prefetch4(A0, xbase);
#pragma unroll
          for (int k = 0; k < 4; ++k) if ((todo >> k) & 1u) prefetch4(B0 + 4 * k, wbase + k * 4096u);
          __builtin_amdgcn_sched_barrier(0);
          if ((todo & 16u) && full4(A0)) todo &= 15u;
#pragma unroll
          for (int k = 0; k < 4; ++k) if (((todo >> k) & 1u) && full4(B0 + 4 * k)) todo &= ~(1u << k);
          if (!ok || todo == 0u) break;
          if (sp.fail(tmo, seq)) ok = false;
        }
        STAMP(29);
      }
      mul(A0, B0, l0);
      STAMP(25);
      finish(0u, 1u, 0u, g0, 0u, 0u, it ? 0u : 1u);
    }
    prefetch4(A1, xbase + 65536u);
    STAMP(2);
    settle4(A1, xbase + 65536u);
    mul(A1, B0, l1);
    finish(1u, 1u, 1u, g0, 0u, 0u, 0u);
    STAMP(3);
    ok = load_frags<16>(B1, rl, wbase + 131072u, lane, ok, 1 + (int)g0, tmo, seq);
    mul(A0, B1, l0);
    prefetch4(A0, xbase + 2u * 65536u);
    mul(A1, B1, l0 + 1024);
    finish(0u, 2u, 0u, g1, 1u, g1, 0u);
    STAMP(4);
    clear(XE_OFF);
    settle4(A0, xbase + 2u * 65536u);
    prefetch4(A1, xbase + 3u * 65536u);
    mul(A0, B0, l1);
    mul(A0, B1, l1 + 1024);
    finish(1u, 2u, 2u, g0, 2u, g1, 0u);
    STAMP(5);
    clear(WE_OFF);
    clear(PE_OFF);
    settle4(A1, xbase + 3u * 65536u);
    VM0();
    mul(A1, B0, l0);
    mul(A1, B1, l0 + 1024);
    finish(0u, 2u, 3u, g0, 3u, g1, 0u);
    STAMP(6);
#pragma unroll
    for (int i = 0; i < 6; ++i) __builtin_amdgcn_raw_buffer_store_b128(zero4, rz, (unsigned)lane * 16u, XS_OFF + i * (1u << 20) + slot, AUX_DEV);
    FLUSH();
  } else if (role >= (unsigned)NPWG) {
    const unsigned wid = (unsigned)(bid - NPWG - NCWG) & 63u;
    const unsigned ks = wid >> 3, q = (wid & 1u) * 16u + (wid & 6u) * 2u + (unsigned)wave;
    u32x2 wt[8];
    {
      const unsigned wbase = (ks * 32u + (lane >> 4) * 8u) * 4096u + ((q & 31u) * 64u + (lane & 15) * 4u) * 2u;
#pragma unroll
      for (int i = 0; i < 8; ++i) wt[i] = u32x2{0u, 0u};
      if (wave < 4) {
#pragma unroll
        for (int i = 0; i < 8; ++i) wt[i] = __builtin_amdgcn_raw_buffer_load_b64(RSRC(pw, IN_BYTES), wbase + i * 4096u, 0, 0);
      }
      __builtin_amdgcn_sched_barrier(0);
    }
    const unsigned seq = __builtin_amdgcn_readfirstlane(seq_v) + 1u;
    const auto rl = RSRC(myws + (seq & 1u) * HALF_BYTES, HALF_BYTES);
    if (wave < 4) {
      u32x4 a[4];
#pragma unroll
      for (int i = 0; i < 4; ++i) {
        const u32x2 lo = wt[2 * i], hi = wt[2 * i + 1];
        a[0][i] = __builtin_amdgcn_perm(hi.x, lo.x, 0x05040100u);
        a[1][i] = __builtin_amdgcn_perm(hi.x, lo.x, 0x07060302u);
        a[2][i] = __builtin_amdgcn_perm(hi.y, lo.y, 0x05040100u);
        a[3][i] = __builtin_amdgcn_perm(hi.y, lo.y, 0x07060302u);
      }
#pragma unroll
      for (int c = 0; c < 4; ++c)
        __builtin_amdgcn_raw_buffer_store_b128(nonzero(a[c]), rl, WE_OFF + ((ks >> 2) * 512u + q * 16u + (ks & 3u) * 4u + c) * 1024u + lane * 16u, 0, AUX_SYS);
    }
    STAMP(1);
    if (lane == 0) __hip_atomic_store(sq, seq, __ATOMIC_RELAXED, __HIP_MEMORY_SCOPE_AGENT);
    if (wid == 0u && wave >= 6)
      __builtin_amdgcn_raw_buffer_store_b128(zero4, RSRC(myws + ((seq & 1u) ^ 1u) * HALF_BYTES, HALF_BYTES), TK_OFF + (unsigned)(wave - 6) * 1024u + lane * 16u, 0, AUX_DEV);
    __builtin_amdgcn_s_sleep(32);
    for (int i = wave; i > 0; --i) __builtin_amdgcn_s_sleep(ZSLEEP / 8 + 1);
    const auto ro = RSRC(outp, OUT_BYTES);
#pragma unroll 1
    for (unsigned e = wid * 8u + (unsigned)wave; e < 6u * CAP; e += 512u) {
      const unsigned f = e / 6u;
      __builtin_amdgcn_raw_buffer_store_b128(zero4, ro, out_off(rank ^ (e - f * 6u + 2u), f) + (lane >> 4) * 4096u + (lane & 15) * 16u, 0, AUX_SYS);
      __builtin_amdgcn_s_sleep(ZSLEEP);
    }
    VM0();
    __syncthreads();
    if (tid == 0) __hip_atomic_fetch_add((gu32*)(myws + (seq & 1u) * HALF_BYTES + CW_OFF), 1u, __ATOMIC_RELAXED, __HIP_MEMORY_SCOPE_AGENT);
    STAMP(2);
    FLUSH();
  } else {
    const unsigned l = ((unsigned)bid / PWL + 1u) & 7u;
    const unsigned pw0 = ((unsigned)bid % PWL) * 8u + (unsigned)wave;
    constexpr unsigned NPW = PWL * 8u, NIN = 512u, LEN = NIN + 1024u, LENP = 2048u, MARK = 1216u, MARKP = 896u;
    const char* peer = P.ws[l];
    const unsigned d = rank ^ l;
    const char* xsrc = peer + XE_OFF + ((rank & 1u) * 2u + (rank >> 2)) * (256u * 1024u);
    const char* wsrc = peer + WE_OFF + ((rank & 1u) * 2u + ((rank >> 1) & 1u)) * (256u * 1024u);
    char* xdst = myws + XS_OFF + l * (256u * 1024u);
    char* tk0 = myws + TK_OFF;
    char* tkp = tk0 + l * 128u;
    const unsigned one = 1u;
    unsigned sv = seq_v;
    const bool w0 = FINE && pw0 >= 32u;
    const char* src0 = w0 ? wsrc + (pw0 - 32u) * 1024u : xsrc + pw0 * 1024u;
    char* dst0 = w0 ? xdst + (WS_OFF - XS_OFF) + (pw0 - 32u) * 1024u : xdst + pw0 * 1024u;
    const char* pout = P.outs[l];
    const char* partner_oe = P.ws[1] + OE_OFF;
    asm volatile("" : "+s"(peer), "+s"(xsrc), "+s"(wsrc), "+s"(xdst), "+s"(tk0), "+s"(tkp), "+s"(src0), "+s"(pout), "+s"(partner_oe));
    const unsigned phase = pw0 * PER / NPW;
    typedef __attribute__((address_space(3))) volatile unsigned lds_vu32;
    lds_vu32* const flow = (lds_vu32*)smem;
    if (tid == 0) flow[0] = 0u;
    auto slot_wait = [&](unsigned stage) {
      const unsigned now = (unsigned)__builtin_amdgcn_s_memrealtime();
      const unsigned left = PER - 1u - (now % PER + PER - phase - 1u) % PER;
      while ((unsigned)__builtin_amdgcn_s_memrealtime() - now < left && __builtin_amdgcn_readfirstlane(flow[0]) < stage) {}
    };
    asm volatile("s_waitcnt lgkmcnt(0)\n\ts_barrier" ::: "memory");
    asm volatile("" : "+v"(sv));
    const unsigned vlin = (unsigned)lane * 16u + ((sv & 1u) ? 0u : HALF_BYTES);
    const unsigned vout = (unsigned)(lane >> 4) * 4096u + (unsigned)(lane & 15) * 16u;
    const unsigned coff = (unsigned)(lane & 7) * 4u + ((sv & 1u) ? 0u : HALF_BYTES);
    const unsigned seq = __builtin_amdgcn_readfirstlane(sv) + 1u;
    gu32* const cwl = (gu32*)(myws + (seq & 1u) * HALF_BYTES + CW_OFF);
    if (lane == 0) __hip_atomic_store(sq, seq, __ATOMIC_RELAXED, __HIP_MEMORY_SCOPE_AGENT);
    u32x4 v; unsigned tk, cw, pg = 0u;
    bool ok = true, cleared = false, planned = false, trusted = false;
    unsigned nn = pw0;
#define TICKET_ASM "s_mov_b64 exec, 1\n\tglobal_atomic_add %1, %[vl], %[one], %[tk] sc0\n\ts_mov_b64 exec, -1"
#define CW_RELOAD() asm volatile("global_load_dword %0, %1, %2 offset:1024 sc1\n\ts_waitcnt vmcnt(0)" : "=&v"(pg) : "v"(coff), "s"(tk0) : "memory")
    auto wait_cleared = [&]() {
      Spin sp;
      while (ok && __builtin_amdgcn_readfirstlane(pg) < (unsigned)NWWG) {
        __builtin_amdgcn_s_sleep(32);
        if (sp.fail(tmo, seq)) ok = false;
        CW_RELOAD();
      }
      cleared = true;
    };
    if (l != 1u) {
      Spin sp;
      unsigned quick = QUICK;
      slot_wait(1u);
      STAMP(3);
      asm volatile("global_load_dwordx4 %0, %[vl], %[src] sc0 sc1\n\t" TICKET_ASM "\n\ts_waitcnt vmcnt(0)"
                   : "=&v"(v), "=&v"(tk) : [vl] "v"(vlin), [src] "s"(src0), [one] "v"(one), [tk] "s"(tkp) : "memory");
      while (ok && !__all(min4(v) != 0u)) {
        slot_wait(quick ? 1u : 9u);
        if (quick && __builtin_amdgcn_readfirstlane(flow[0])) --quick;
        asm volatile("global_load_dwordx4 %0, %1, %2 sc0 sc1\n\ts_waitcnt vmcnt(0)" : "=&v"(v) : "v"(vlin), "s"(src0) : "memory");
        if (sp.fail(tmo, seq)) ok = false;
      }
      nn = __builtin_amdgcn_readfirstlane(tk) + NPW;
      if (!ok) v = nanv;
      asm volatile("global_store_dwordx4 %0, %1, %2 sc0 sc1\n\ts_nop 1" :: "v"(vlin), "v"(v), "s"(dst0) : "memory");
      if (__builtin_amdgcn_readfirstlane(flow[0]) == 0u) flow[0] = 1u;
    } else {
      const unsigned vidle = (unsigned)lane * 16u + ((sv & 1u) ? HALF_BYTES : 0u);
      bool first = true;
      u32x4 a0, a1;
#pragma unroll 1
      for (;;) {
        const unsigned f = nn;
        const char* const src = (f < NPUSH ? myws : peer) + PE_OFF + f * 1024u;
        const char* const fsrc = myws + FE_OFF + f * 2048u;
        const char* const cwp = !cleared && f >= 256u ? tk0 : tk0 + 128;
        asm volatile("global_load_dwordx4 %0, %[vl], %[src] sc0 sc1\n\tglobal_load_dwordx4 %3, %[vl], %[fs] sc1\n\t"
                     "global_load_dwordx4 %4, %[vl], %[fs] offset:1024 sc1\n\tglobal_load_dword %2, %[co], %[cp] offset:1024 sc1\n\t" TICKET_ASM
                     : "=&v"(v), "=&v"(tk), "=&v"(cw), "=&v"(a0), "=&v"(a1)
                     : [vl] "v"(vlin), [src] "s"(src), [fs] "s"(fsrc), [co] "v"(coff), [cp] "s"(cwp), [one] "v"(one), [tk] "s"(tkp) : "memory");
        if (f == MARKP) __hip_atomic_store(cwl + 1, (unsigned)__builtin_amdgcn_s_memrealtime() | 1u, __ATOMIC_RELAXED, __HIP_MEMORY_SCOPE_AGENT);
        const char* const odst = myws + OE_OFF + f * 1024u;
        const char* const ddst = outp + out_off(rank, f);
        asm volatile("s_waitcnt vmcnt(0)\n\tv_mov_b32 %4, %5" : "+v"(v), "+v"(tk), "+v"(a0), "+v"(a1), "=&v"(pg) : "v"(cw) : "memory");
        if (!__all(min(min(min4(v), min4(a0)), min4(a1)) != 0u)) {
          Spin sp;
          unsigned quick = QUICKP;
          while (ok) {
            if (f < NPUSH) __builtin_amdgcn_s_sleep(2);
            else if (quick) --quick;
            else slot_wait(9u);
            asm volatile("global_load_dwordx4 %0, %3, %4 sc0 sc1\n\tglobal_load_dwordx4 %1, %3, %5 sc1\n\tglobal_load_dwordx4 %2, %3, %5 offset:1024 sc1\n\ts_waitcnt vmcnt(0)"
                         : "=&v"(v), "=&v"(a0), "=&v"(a1) : "v"(vlin), "s"(src), "s"(fsrc) : "memory");
            if (__all(min(min(min4(v), min4(a0)), min4(a1)) != 0u)) break;
            if (sp.fail(tmo, seq)) ok = false;
          }
        }
        nn = __builtin_amdgcn_readfirstlane(tk) + NPW;
        if (!cleared && f >= 256u) wait_cleared();
        if (first) { asm volatile("global_store_dwordx4 %0, %1, %2 sc0 sc1\n\ts_waitcnt vmcnt(0)" :: "v"(vidle), "v"(zero4), "s"(odst) : "memory"); first = false; }
        f32x4 s0 = __builtin_bit_cast(f32x4, a0), s1 = __builtin_bit_cast(f32x4, a1);
        s0 += f32x4{__builtin_bit_cast(float, v[0] << 16), __builtin_bit_cast(float, v[0] & 0xffff0000u), __builtin_bit_cast(float, v[1] << 16), __builtin_bit_cast(float, v[1] & 0xffff0000u)};
        s1 += f32x4{__builtin_bit_cast(float, v[2] << 16), __builtin_bit_cast(float, v[2] & 0xffff0000u), __builtin_bit_cast(float, v[3] << 16), __builtin_bit_cast(float, v[3] & 0xffff0000u)};
        const u32x4 o = ok ? nonzero(to_bf16x8(s0, s1)) : nanv;
        asm volatile("global_store_dwordx4 %0, %2, %3 sc0 sc1\n\tglobal_store_dwordx4 %1, %2, %4 sc1\n\ts_nop 1" :: "v"(vlin), "v"(vout), "v"(o), "s"(odst), "s"(ddst) : "memory");
        if (nn >= 1024u) break;
        asm volatile("global_store_dwordx4 %0, %1, %2 sc0 sc1\n\ts_nop 1" :: "v"(vidle), "v"(zero4), "s"(myws + OE_OFF + nn * 1024u) : "memory");
      }
      cleared = true;
    }
    unsigned lim = l == 1u ? LENP : LEN, lo = 0u;
    auto plan = [&]() {
      const bool on = lane >= 1 && lane < NR;
      const int rel = on ? (int)(pg - __builtin_amdgcn_readlane(pg, 1)) : 0;
      const int e = on ? (max(min(rel, 4000), -4000) * RATE >> 8) + (lane == 1 ? (int)(LENP - MARKP) + PBIAS : (int)(LEN - MARK)) : 0;
      int sum = 0;
#pragma unroll
      for (int i = 1; i < NR; ++i) sum += __builtin_amdgcn_readlane(e, i);
      const bool act = on && (lane == 1 || e * 7 < sum);
      const int ea = act ? e : 0;
      sum = 0;
#pragma unroll
      for (int i = 1; i < NR; ++i) sum += __builtin_amdgcn_readlane(ea, i);
      const int level = sum / (int)__popcll(__ballot(act));
      const unsigned h = act && lane != 1 ? (unsigned)max(level - e, 0) : 0u;
      unsigned H = 0u, hi = 0u;
#pragma unroll
      for (int i = 2; i < NR; ++i) {
        if ((unsigned)i == l) lo = H;
        H = min(H + __builtin_amdgcn_readlane(h, i), CAP);
        if ((unsigned)i == l) hi = H;
      }
      lim = l == 1u ? LENP - H : LEN + (hi - lo);
      planned = true;
    };
#pragma unroll 1
    for (;;) {
      if (!planned && nn >= (l == 1u ? LENP - CAP : LEN)) {
        Spin sp;
        while (ok && !__all(pg != 0u)) {
          __builtin_amdgcn_s_sleep(16);
          if (sp.fail(tmo, seq)) ok = false;
          CW_RELOAD();
        }
        if (ok) plan();
        planned = true;
      }
      if (nn >= lim) break;
      const unsigned n = nn;
      const char* src; const char* direct; char* dst;
      bool relay = false;
      if (l != 1u && n < NIN) {
        const unsigned m = FINE && n < 128u ? (n < 96u ? n + 32u : n - 64u) : n;
        const unsigned o = (m >> 7) * 65536u + (m & 63u) * 1024u;
        const bool isw = (0x56u >> (m >> 6)) & 1u;
        src = (isw ? wsrc : xsrc) + o; dst = xdst + (isw ? WS_OFF - XS_OFF : 0u) + o;
        direct = src;
      } else if (l != 1u && n >= LEN) {
        const unsigned f = (lo + (n - LEN)) & 1023u, o = out_off(rank ^ 1u, f);
        direct = partner_oe + f * 1024u;
        relay = trusted && ok;
        src = relay ? pout + o : direct; dst = outp + o;
      } else {
        const unsigned m = n - 1024u;
        const unsigned f = (l != 1u ? n - NIN : m < 1024u - CAP ? CAP + m : 1023u - m) & 1023u;
        src = peer + OE_OFF + f * 1024u; dst = outp + out_off(d, f);
        direct = src;
      }
      const bool isout = l == 1u || n >= NIN;
      const unsigned vsrc = relay ? vout : vlin, vdst = isout ? vout : vlin;
      const char* const cwp = (!cleared && isout) || (!planned && n >= (l == 1u ? MARKP : MARK) + 2u * NPW) ? tk0 : tk0 + 128;
      asm volatile("global_load_dwordx4 %0, %[vs], %[src] sc0 sc1\n\tglobal_load_dword %2, %[co], %[cp] offset:1024 sc1\n\t" TICKET_ASM
                   : "=&v"(v), "=&v"(tk), "=&v"(cw)
                   : [vs] "v"(vsrc), [vl] "v"(vlin), [src] "s"(src), [co] "v"(coff), [cp] "s"(cwp), [one] "v"(one), [tk] "s"(tkp) : "memory");
      if (l != 1u && n == MARK) __hip_atomic_store(cwl + l, (unsigned)__builtin_amdgcn_s_memrealtime() | 1u, __ATOMIC_RELAXED, __HIP_MEMORY_SCOPE_AGENT);
      if (!planned && __all(pg != 0u)) plan();
      asm volatile("s_waitcnt vmcnt(0)\n\tv_mov_b32 %2, %3" : "+v"(v), "+v"(tk), "=&v"(pg) : "v"(cw) : "memory");
      if (!__all(min4(v) != 0u)) {
        Spin sp;
        while (ok) {
          if (isout || sp.it >= QUICK) slot_wait(9u);
          asm volatile("global_load_dwordx4 %0, %1, %2 sc0 sc1\n\ts_waitcnt vmcnt(0)" : "=&v"(v) : "v"(vlin), "s"(direct) : "memory");
          if (__all(min4(v) != 0u)) break;
          if (sp.fail(tmo, seq)) ok = false;
        }
      }
      nn = __builtin_amdgcn_readfirstlane(tk) + NPW;
      if (!cleared && isout) wait_cleared();
      if (!ok) v = nanv;
      asm volatile("global_store_dwordx4 %0, %1, %2 sc0 sc1\n\ts_nop 1" :: "v"(vdst), "v"(v), "s"(dst) : "memory");
      trusted = trusted || (ok && n >= NIN + 256u && n < LEN);
    }
    STAMP(7);
    FLUSH();
  }
}

}

extern "C" void kernel_launch(int rank, int nranks, ncclComm_t comm,
                              hipStream_t stream,
                              void* const* d_in, const int* in_sizes, int n_in,
                              void* d_out, int out_size,
                              const int* in_dtypes, int out_dtype,
                              void* const* d_out_all,
                              void* const* d_ws_all, size_t ws_bytes) {
  if (nranks != NR || (unsigned)rank >= (unsigned)NR || n_in < 2 || ws_bytes < WS_USED ||
      (size_t)out_size * 2 != OUT_BYTES || (size_t)in_sizes[0] * 2 != IN_BYTES || (size_t)in_sizes[1] * 2 != IN_BYTES) {
    fprintf(stderr, "kernel_launch: unexpected problem shape\n");
    return;
  }
  Params P;
  for (int i = 0; i < NR; ++i) { P.ws[i] = (char*)d_ws_all[rank ^ i]; P.outs[i] = (char*)d_out_all[rank ^ i]; }
  P.out = (char*)d_out;
  P.x = (const char*)d_in[0];
  P.w = (const char*)d_in[1];
  P.rank = rank;
  P.ws0 = P.ws[0];
  P.pad = 0;
  for (int i = 0; i < 6; ++i) P.p[i] = 0;
  gemm_ar_kernel<<<dim3(NWG), dim3(512), 0, stream>>>(P);
  HIP_CHECK(hipGetLastError());
}
```

```cpp
#include <hip/hip_runtime.h>
#include <rccl/rccl.h>
#include <cstdio>
#include <cstdlib>

#define HIP_CHECK(expr) do { hipError_t _e = (expr); if (_e != hipSuccess) { \
    fprintf(stderr, "HIP error %s at %s:%d\n", hipGetErrorString(_e), __FILE__, __LINE__); \
    return; } } while (0)

namespace {

typedef unsigned u32x4 __attribute__((ext_vector_type(4)));
typedef unsigned u32x2 __attribute__((ext_vector_type(2)));
typedef float f32x4 __attribute__((ext_vector_type(4)));
typedef __bf16 bf16x8 __attribute__((ext_vector_type(8)));
typedef __attribute__((address_space(1))) unsigned gu32;

constexpr unsigned PER = 107;
constexpr int RATE = 153;
constexpr int PBIAS = 12;
constexpr unsigned QUICK = 2;
constexpr unsigned NPUSH = 64;
constexpr int ZSLEEP = 12;
constexpr int NR = 8;
constexpr int PWL = 8;
constexpr int NPWG = 7 * PWL;
constexpr int NSET = 16;
constexpr int NCWG = 8 * NSET;
constexpr int NWWG = 64;
constexpr int NWG = NWWG + NCWG + NPWG;
constexpr unsigned IN_BYTES = 2048u * 256u * 2u;
constexpr unsigned OUT_BYTES = 2048u * 2048u * 2u;
constexpr unsigned XS_OFF = 0;
constexpr unsigned WS_OFF = XS_OFF + (2u << 20);
constexpr unsigned FE_OFF = WS_OFF + (2u << 20);
constexpr unsigned XE_OFF = FE_OFF + (2u << 20);
constexpr unsigned WE_OFF = XE_OFF + (1u << 20);
constexpr unsigned PE_OFF = WE_OFF + (1u << 20);
constexpr unsigned OE_OFF = PE_OFF + (1u << 20);
constexpr unsigned TK_OFF = OE_OFF + (1u << 20);
constexpr unsigned CW_OFF = TK_OFF + 8 * 128;
constexpr unsigned HALF_BYTES = TK_OFF + 4096;
constexpr unsigned SQ_OFF = 2 * HALF_BYTES;
constexpr unsigned TMO_OFF = SQ_OFF + 256 * 64;
constexpr unsigned WS_USED = TMO_OFF + 128;
constexpr unsigned long long DEADLINE_TICKS = 50000000ull;
constexpr unsigned NANS = 0x7fc07fc0u;
constexpr unsigned long long REG_TC = 0xFEBA7632DC985410ull;
constexpr unsigned CAP = 896;

constexpr int AUX_SYS = 17;
constexpr int AUX_DEV = 16;

struct Params {
  const char* x;
  const char* w;
  char* ws0;
  int rank;
  int pad;
  char* out;
  int pad2[6];
  char* ws[NR];
  char* outs[NR];
};

#define RSRC(p, n) __builtin_amdgcn_make_buffer_rsrc((void*)(p), 0, (int)(n), 0x00020000)
#define VM0() asm volatile("s_waitcnt vmcnt(0)" ::: "memory")

__device__ __forceinline__ unsigned min4(u32x4 v) { return min(min(min(v[0], v[1]), v[2]), v[3]); }
__device__ __forceinline__ u32x4 nonzero(u32x4 v) {
#pragma unroll
  for (int i = 0; i < 4; ++i) v[i] = v[i] ? v[i] : 0x80000000u;
  return v;
}
__device__ __forceinline__ u32x4 to_bf16x8(f32x4 a, f32x4 b) {
  bf16x8 o;
#pragma unroll
  for (int j = 0; j < 4; ++j) { o[j] = (__bf16)a[j]; o[4 + j] = (__bf16)b[j]; }
  return __builtin_bit_cast(u32x4, o);
}
__device__ __forceinline__ unsigned out_off(unsigned o, unsigned f) {
  if (f < 128u) f = (f & 67u) + ((f & 28u) << 1) + ((f & 32u) >> 3);
  const unsigned tc = (unsigned)(REG_TC >> ((f >> 6) * 4u)) & 15u;
  const unsigned row = (o >> 2) * 1024u + (tc >> 2) * 256u + ((f >> 3) & 7u) * 32u + (o & 1u) * 16u + (f & 3u) * 4u;
  return row * 4096u + ((o >> 1) & 1u) * 2048u + (tc & 3u) * 512u + ((f >> 2) & 1u) * 256u;
}

struct Spin {
  unsigned it = 0; unsigned long long t0 = 0;
  __device__ __forceinline__ bool fail(gu32* tmo, unsigned seq) {
    asm volatile("" ::: "memory");
    if (it == 0) t0 = __builtin_amdgcn_s_memrealtime();
    if ((++it & 63u) != 0u) return false;
    const unsigned t = __hip_atomic_load(tmo, __ATOMIC_RELAXED, __HIP_MEMORY_SCOPE_AGENT);
    if (__builtin_amdgcn_readfirstlane(t) == seq || __builtin_amdgcn_s_memrealtime() - t0 > DEADLINE_TICKS) {
      __hip_atomic_store(tmo, seq, __ATOMIC_RELAXED, __HIP_MEMORY_SCOPE_AGENT);
      return true;
    }
    return false;
  }
};

__global__ void __launch_bounds__(512, 2) gemm_ar_kernel(Params P) {
  __shared__ __attribute__((aligned(16))) float smem[2 * 8 * 2048 + 64];
  const int tid = threadIdx.x;
  const int lane = tid & 63;
  const int wave = __builtin_amdgcn_readfirstlane(tid >> 6);
  const int bid = blockIdx.x;
  const char* px = P.x; const char* pw = P.w; char* myws = P.ws0; unsigned rank = (unsigned)P.rank & 7u; char* outp = P.out;
  const unsigned cid = (unsigned)(bid - NPWG);
  const unsigned g = (cid & 1u) * 64u + (cid & 6u) * 8u + (cid >> 3);
  unsigned xoff = cid < (unsigned)NCWG ? (g * 16u + (lane & 15)) * 512u + (unsigned)wave * 64u + (lane >> 4) * 16u : IN_BYTES;
  unsigned sqoff = SQ_OFF + bid * 64 + wave * 4;
  asm volatile("" : "+v"(xoff), "+s"(sqoff));
  asm volatile("" : "+s"(px), "+s"(pw), "+s"(myws), "+s"(rank), "+s"(outp));
  const u32x4 xv = __builtin_amdgcn_raw_buffer_load_b128(RSRC(px, IN_BYTES), xoff, 0, 0);
  gu32* const sq = (gu32*)(myws + sqoff);
  gu32* const tmo = (gu32*)(myws + TMO_OFF);
  const unsigned seq_v = __hip_atomic_load(sq, __ATOMIC_RELAXED, __HIP_MEMORY_SCOPE_AGENT);
  const u32x4 nanv = {NANS, NANS, NANS, NANS};
  const u32x4 zero4 = {0u, 0u, 0u, 0u};
  __builtin_amdgcn_sched_barrier(0);
  if (cid < (unsigned)NCWG) {
    unsigned xe = XE_OFF + (((unsigned)wave >> 2) * 512u + g * 4u + ((unsigned)wave & 3u)) * 1024u + lane * 16u;
    asm volatile("" : "+v"(xe));
    __builtin_amdgcn_raw_buffer_store_b128(nonzero(xv), RSRC(myws, 2 * HALF_BYTES), xe + ((seq_v & 1u) ? 0u : HALF_BYTES), 0, AUX_SYS);
  }
  unsigned role = (unsigned)bid;
  asm volatile("" : "+s"(role));

  if (role - NPWG < (unsigned)NCWG) {
    typedef __attribute__((address_space(3))) volatile unsigned lds_vu32;
    lds_vu32* const ctl = (lds_vu32*)(smem + 2 * 8 * 2048);
    const unsigned s = cid & 7u, set = cid >> 3, v = (unsigned)wave;
    const unsigned slot = (cid * 8u + v) * 1024u;
    const unsigned seq = __builtin_amdgcn_readfirstlane(seq_v) + 1u;
    const auto rl = RSRC(myws + (seq & 1u) * HALF_BYTES, HALF_BYTES);
    const auto rz = RSRC(myws + ((seq & 1u) ^ 1u) * HALF_BYTES, HALF_BYTES);
    if (lane == 0) __hip_atomic_store(sq, seq, __ATOMIC_RELAXED, __HIP_MEMORY_SCOPE_AGENT);
    const auto rp = RSRC(P.ws[1] + (seq & 1u) * HALF_BYTES, HALF_BYTES);
    if (tid == 0) ctl[0] = 0u;
    asm volatile("s_waitcnt lgkmcnt(0)\n\ts_barrier" ::: "memory");
    bool ok = true;
    const unsigned xbase = (v < 2u ? XE_OFF + (v * 2u + (rank >> 2)) * (256u * 1024u) : XS_OFF + v * (256u * 1024u)) + set * 4096u;
    const unsigned wbase = (v < 2u ? WE_OFF + (v * 2u + ((rank >> 1) & 1u)) * (256u * 1024u) : WS_OFF + v * (256u * 1024u)) + ((s & 1u) * 4u + (s >> 1)) * 16384u;
    const bool own = (set & 1u) == (rank & 1u);
    const unsigned vl = (unsigned)lane * 16u;
    u32x4 A[4], B[16];
#pragma unroll
    for (int i = 0; i < 4; ++i) asm volatile("" : "=v"(A[i]), "=v"(B[i]), "=v"(B[4 + i]), "=v"(B[8 + i]), "=v"(B[12 + i]));
    const char* const lbase = myws + (seq & 1u) * HALF_BYTES;
    auto fetch4 = [&](u32x4* F, unsigned off, unsigned on) {
      asm volatile("s_cmp_eq_u32 %6, 0\n\ts_cbranch_scc1 1f\n\t"
                   "global_load_dwordx4 %0, %4, %5 sc0 sc1\n\tglobal_load_dwordx4 %1, %4, %5 offset:1024 sc0 sc1\n\t"
                   "global_load_dwordx4 %2, %4, %5 offset:2048 sc0 sc1\n\tglobal_load_dwordx4 %3, %4, %5 offset:3072 sc0 sc1\n1:"
                   : "+v"(F[0]), "+v"(F[1]), "+v"(F[2]), "+v"(F[3]) : "v"(vl), "s"(lbase + off), "s"(on) : "memory", "scc");
    };
    auto landed4 = [&](u32x4* F) { asm volatile("" : "+v"(F[0]), "+v"(F[1]), "+v"(F[2]), "+v"(F[3])); };
    auto full4 = [&](const u32x4* F) { return __all(min(min(min4(F[0]), min4(F[1])), min(min4(F[2]), min4(F[3]))) != 0u); };
    const unsigned islot = ((lane >> 4) * 16u + ((s >> 1) & 1u) * 8u + ((lane & 15) >> 1)) * 16u;
    const unsigned vst = own ? islot + (lane & 1) * 1024u : (lane & 1) ? HALF_BYTES : islot;
    unsigned young = 0u;
#pragma unroll 1
    for (unsigned p = v < 2u ? 1u : 0u; p < 9u; ++p) {
      const unsigned t = (p - 1u) & 3u, h = p >= 5u ? 1u : 0u;
      if (young == 2u) asm volatile("s_waitcnt vmcnt(2)" ::: "memory");
      else if (young == 1u) asm volatile("s_waitcnt vmcnt(1)" ::: "memory");
      else VM0();
      landed4(A);
      unsigned todo = (p == 1u ? 31u : p == 5u ? 15u : 0u) | (p >= 2u && !full4(A) ? 16u : 0u);
      {
        const unsigned boff = wbase + h * 131072u, aoff = xbase + t * 65536u;
        Spin sp;
        while (ok && todo && !__all(__builtin_amdgcn_raw_buffer_load_b32(rl, todo & 15u ? 8188u : 2044u, todo & 15u ? boff : aoff, AUX_SYS) != 0u)) {
          __builtin_amdgcn_s_sleep(2);
          if (sp.fail(tmo, seq)) ok = false;
        }
#pragma clang loop unroll(disable)
        do {
          asm volatile("" : "+s"(sp.it));
          fetch4(A, aoff, todo & 16u);
#pragma unroll
          for (int k = 0; k < 4; ++k) fetch4(B + 4 * k, boff + k * 4096u, todo & (1u << k));
          const unsigned anyb = todo & 15u;
          if (anyb == 15u) asm volatile("s_waitcnt vmcnt(12)" ::: "memory"); else if (todo) VM0();
          landed4(A); landed4(B);
          if ((todo & 16u) && full4(A)) todo &= 15u;
          if (anyb && full4(B)) todo &= ~1u;
          asm volatile("s_waitcnt vmcnt(8)" ::: "memory"); landed4(B + 4);
          if (anyb && full4(B + 4)) todo &= ~2u;
          asm volatile("s_waitcnt vmcnt(4)" ::: "memory"); landed4(B + 8);
          if (anyb && full4(B + 8)) todo &= ~4u;
          if (anyb) VM0();
          landed4(B + 12);
          if (anyb && full4(B + 12)) todo &= ~8u;
          if (todo && sp.fail(tmo, seq)) ok = false;
        } while (ok && todo);
      }
      if (p == 8u) VM0();
      float* const buf = smem + (p & 1u) * 8192u;
      {
        f32x4 acc[4] = {};
#pragma unroll
        for (int ks = 0; ks < 4; ++ks) {
#pragma unroll
          for (int c = 0; c < 4; ++c)
            acc[c] = __builtin_amdgcn_mfma_f32_16x16x32_bf16(__builtin_bit_cast(bf16x8, A[ks]), __builtin_bit_cast(bf16x8, B[ks * 4 + c]), acc[c], 0, 0, 0);
        }
        __builtin_amdgcn_sched_barrier(0);
        fetch4(A, xbase + (p & 3u) * 65536u, (0xFEu >> p) & 1u);
        float* const dst = buf + v * 1024u + (lane >> 4) * 256 + (lane & 15) * 4;
#pragma unroll
        for (int r = 0; r < 4; ++r) *(f32x4*)(dst + r * 64) = f32x4{acc[0][r], acc[1][r], acc[2][r], acc[3][r]};
      }
      if (!ok) ctl[0] = 1u;
      if (p) __syncthreads();
      young = 0u;
      if ((v >> 2) != (p & 1u)) {
        const float* src = buf + (v & 3u) * 256u + lane * 4;
        f32x4 r[8];
        const bool bad = ctl[0] != 0u;
#pragma unroll
        for (int j = 0; j < 8; ++j) r[j] = *(const f32x4*)(src + j * 1024);
        __builtin_amdgcn_sched_barrier(0);
        const f32x4 sum = ((r[0] + r[1]) + (r[2] + r[3])) + ((r[4] + r[5]) + (r[6] + r[7]));
        u32x4 o = __builtin_bit_cast(u32x4, sum);
        if (!own) {
          const u32x4 hb = to_bf16x8(sum, sum);
          o = u32x4{hb[0], hb[1], (unsigned)__builtin_amdgcn_update_dpp(0, (int)hb[0], 0xF5, 0xF, 0xF, true), (unsigned)__builtin_amdgcn_update_dpp(0, (int)hb[1], 0xF5, 0xF, 0xF, true)};
        }
        const unsigned reg = h * 8u + t * 2u + (s & 1u);
        const unsigned f = reg * 64u + (reg < 2u ? (s >> 2) * 32u + (set >> 1) * 4u : (set >> 1) * 8u + (s >> 2) * 4u) + (v & 3u);
        o = bad ? nanv : nonzero(o);
        asm volatile("" : "+v"(o));
        if (p) {
          __builtin_amdgcn_raw_buffer_store_b128(o, !own && reg * 64u < NPUSH ? rp : rl, vst, own ? FE_OFF + f * 2048u : PE_OFF + f * 1024u, AUX_SYS);
          young = 1u;
        }
      }
      if (p - 5u < 3u) { __builtin_amdgcn_raw_buffer_store_b128(zero4, rz, vl, XE_OFF + (p - 5u) * (1u << 20) + slot, AUX_SYS); ++young; }
    }
#pragma unroll
    for (int i = 0; i < 6; ++i) __builtin_amdgcn_raw_buffer_store_b128(zero4, rz, (unsigned)lane * 16u, XS_OFF + i * (1u << 20) + slot, AUX_DEV);
  } else if (role >= (unsigned)NPWG) {
    const unsigned wid = (unsigned)(bid - NPWG - NCWG) & 63u;
    const unsigned ks = wid >> 3, q = (wid & 1u) * 16u + (wid & 6u) * 2u + (unsigned)wave;
    u32x2 wt[8];
    {
      const unsigned wbase = (ks * 32u + (lane >> 4) * 8u) * 4096u + ((q & 31u) * 64u + (lane & 15) * 4u) * 2u;
#pragma unroll
      for (int i = 0; i < 8; ++i) wt[i] = u32x2{0u, 0u};
      if (wave < 4) {
#pragma unroll
        for (int i = 0; i < 8; ++i) wt[i] = __builtin_amdgcn_raw_buffer_load_b64(RSRC(pw, IN_BYTES), wbase + i * 4096u, 0, 0);
      }
      __builtin_amdgcn_sched_barrier(0);
    }
    const unsigned seq = __builtin_amdgcn_readfirstlane(seq_v) + 1u;
    const auto rl = RSRC(myws + (seq & 1u) * HALF_BYTES, HALF_BYTES);
    if (wave < 4) {
      u32x4 a[4];
#pragma unroll
      for (int i = 0; i < 4; ++i) {
        const u32x2 lo = wt[2 * i], hi = wt[2 * i + 1];
        a[0][i] = __builtin_amdgcn_perm(hi.x, lo.x, 0x05040100u);
        a[1][i] = __builtin_amdgcn_perm(hi.x, lo.x, 0x07060302u);
        a[2][i] = __builtin_amdgcn_perm(hi.y, lo.y, 0x05040100u);
        a[3][i] = __builtin_amdgcn_perm(hi.y, lo.y, 0x07060302u);
      }
#pragma unroll
      for (int c = 0; c < 4; ++c)
        __builtin_amdgcn_raw_buffer_store_b128(nonzero(a[c]), rl, WE_OFF + ((ks >> 2) * 512u + q * 16u + (ks & 3u) * 4u + c) * 1024u + lane * 16u, 0, AUX_SYS);
    }
    if (lane == 0) __hip_atomic_store(sq, seq, __ATOMIC_RELAXED, __HIP_MEMORY_SCOPE_AGENT);
    if (wid == 0u && wave >= 6)
      __builtin_amdgcn_raw_buffer_store_b128(zero4, RSRC(myws + ((seq & 1u) ^ 1u) * HALF_BYTES, HALF_BYTES), TK_OFF + (unsigned)(wave - 6) * 1024u + lane * 16u, 0, AUX_DEV);
    __builtin_amdgcn_s_sleep(32);
    for (int i = wave; i > 0; --i) __builtin_amdgcn_s_sleep(ZSLEEP / 8 + 1);
    const auto ro = RSRC(outp, OUT_BYTES);
#pragma unroll 1
    for (unsigned e = wid * 8u + (unsigned)wave; e < 6u * CAP; e += 512u) {
      const unsigned f = e / 6u;
      __builtin_amdgcn_raw_buffer_store_b128(zero4, ro, out_off(rank ^ (e - f * 6u + 2u), f) + (lane >> 4) * 4096u + (lane & 15) * 16u, 0, AUX_SYS);
      __builtin_amdgcn_s_sleep(ZSLEEP);
    }
    VM0();
    __syncthreads();
    if (tid == 0) __hip_atomic_fetch_add((gu32*)(myws + (seq & 1u) * HALF_BYTES + CW_OFF), 1u, __ATOMIC_RELAXED, __HIP_MEMORY_SCOPE_AGENT);
  } else {
    const unsigned l = ((unsigned)bid / PWL + 1u) & 7u;
    const unsigned pw0 = ((unsigned)bid % PWL) * 8u + (unsigned)wave;
    constexpr unsigned NPW = PWL * 8u, NIN = 512u, LEN = NIN + 1024u, LENP = 2048u, MARK = 960u, MARKP = 640u, POLLN = 1344u;
    const char* peer = P.ws[l];
    const unsigned d = rank ^ l;
    const char* xsrc = peer + XE_OFF + ((rank & 1u) * 2u + (rank >> 2)) * (256u * 1024u);
    const char* wsrc = peer + WE_OFF + ((rank & 1u) * 2u + ((rank >> 1) & 1u)) * (256u * 1024u);
    char* xdst = myws + XS_OFF + l * (256u * 1024u);
    char* tk0 = myws + TK_OFF;
    char* tkp = tk0 + l * 128u;
    const unsigned one = 1u;
    unsigned sv = seq_v;
    const char* src0 = xsrc + pw0 * 1024u;
    char* const dst0 = xdst + pw0 * 1024u;
    const char* pout = P.outs[l];
    const char* partner_oe = P.ws[1] + OE_OFF;
    asm volatile("" : "+s"(peer), "+s"(xsrc), "+s"(wsrc), "+s"(xdst), "+s"(tk0), "+s"(tkp), "+s"(src0), "+s"(pout), "+s"(partner_oe));
    const unsigned phase = pw0 * PER / NPW;
    typedef __attribute__((address_space(3))) volatile unsigned lds_vu32;
    lds_vu32* const flow = (lds_vu32*)smem;
    if (tid == 0) flow[0] = 0u;
    auto slot_wait = [&](unsigned stage) {
      const unsigned now = (unsigned)__builtin_amdgcn_s_memrealtime();
      const unsigned left = PER - 1u - (now % PER + PER - phase - 1u) % PER;
      while ((unsigned)__builtin_amdgcn_s_memrealtime() - now < left && __builtin_amdgcn_readfirstlane(flow[0]) < stage) {}
    };
    asm volatile("s_waitcnt lgkmcnt(0)\n\ts_barrier" ::: "memory");
    asm volatile("" : "+v"(sv));
    const unsigned vlin = (unsigned)lane * 16u + ((sv & 1u) ? 0u : HALF_BYTES);
    const unsigned vout = (unsigned)(lane >> 4) * 4096u + (unsigned)(lane & 15) * 16u;
    const unsigned coff = (unsigned)(lane & 7) * 4u + ((sv & 1u) ? 0u : HALF_BYTES);
    const unsigned seq = __builtin_amdgcn_readfirstlane(sv) + 1u;
    gu32* const cwl = (gu32*)(myws + (seq & 1u) * HALF_BYTES + CW_OFF);
    if (lane == 0) __hip_atomic_store(sq, seq, __ATOMIC_RELAXED, __HIP_MEMORY_SCOPE_AGENT);
    u32x4 v; unsigned tk, cw, pg = 0u;
    bool ok = true, cleared = false, planned = false, trusted = false;
    unsigned nn = pw0;
#define TICKET_ASM "s_mov_b64 exec, 1\n\tglobal_atomic_add %1, %[vl], %[one], %[tk] sc0\n\ts_mov_b64 exec, -1"
#define CW_RELOAD() asm volatile("global_load_dword %0, %1, %2 offset:1024 sc1\n\ts_waitcnt vmcnt(0)" : "=&v"(pg) : "v"(coff), "s"(tk0) : "memory")
    auto wait_cleared = [&]() {
      Spin sp;
      while (ok && __builtin_amdgcn_readfirstlane(pg) < (unsigned)NWWG) {
        __builtin_amdgcn_s_sleep(32);
        if (sp.fail(tmo, seq)) ok = false;
        CW_RELOAD();
      }
      cleared = true;
    };
    if (l != 1u && wave != 7) {
      Spin sp;
      unsigned quick = QUICK;
      slot_wait(1u);
      asm volatile("global_load_dwordx4 %0, %[vl], %[src] sc0 sc1\n\t" TICKET_ASM "\n\ts_waitcnt vmcnt(0)"
                   : "=&v"(v), "=&v"(tk) : [vl] "v"(vlin), [src] "s"(src0), [one] "v"(one), [tk] "s"(tkp) : "memory");
      while (ok && !__all(min4(v) != 0u)) {
        slot_wait(quick ? 1u : 9u);
        if (quick && __builtin_amdgcn_readfirstlane(flow[0])) --quick;
        asm volatile("global_load_dwordx4 %0, %1, %2 sc0 sc1\n\ts_waitcnt vmcnt(0)" : "=&v"(v) : "v"(vlin), "s"(src0) : "memory");
        if (sp.fail(tmo, seq)) ok = false;
      }
      nn = __builtin_amdgcn_readfirstlane(tk) + NPW;
      if (!ok) v = nanv;
      asm volatile("global_store_dwordx4 %0, %1, %2 sc0 sc1\n\ts_nop 1" :: "v"(vlin), "v"(v), "s"(dst0) : "memory");
      if (__builtin_amdgcn_readfirstlane(flow[0]) == 0u) flow[0] = 1u;
    } else if (l == 1u) {
      const unsigned vidle = (unsigned)lane * 16u + ((sv & 1u) ? HALF_BYTES : 0u);
      bool first = true, dry = true;
      u32x4 a0, a1;
#pragma unroll 1
      for (;;) {
        const unsigned f = nn;
        const char* const src = (f < NPUSH ? myws : peer) + PE_OFF + f * 1024u;
        const char* const fsrc = myws + FE_OFF + f * 2048u;
        const char* const cwp = !cleared && f >= 256u ? tk0 : tk0 + 128;
        asm volatile("global_load_dwordx4 %0, %[vl], %[src] sc0 sc1\n\tglobal_load_dwordx4 %3, %[vl], %[fs] sc1\n\t"
                     "global_load_dwordx4 %4, %[vl], %[fs] offset:1024 sc1\n\tglobal_load_dword %2, %[co], %[cp] offset:1024 sc1\n\t" TICKET_ASM
                     : "=&v"(v), "=&v"(tk), "=&v"(cw), "=&v"(a0), "=&v"(a1)
                     : [vl] "v"(vlin), [src] "s"(src), [fs] "s"(fsrc), [co] "v"(coff), [cp] "s"(cwp), [one] "v"(one), [tk] "s"(dry ? tk0 : tkp) : "memory");
        if (f == MARKP) __hip_atomic_store(cwl + 1, (unsigned)__builtin_amdgcn_s_memrealtime() | 1u, __ATOMIC_RELAXED, __HIP_MEMORY_SCOPE_AGENT);
        const char* const odst = myws + OE_OFF + f * 1024u;
        const char* const ddst = outp + out_off(rank, f);
        asm volatile("s_waitcnt vmcnt(0)\n\tv_mov_b32 %4, %5" : "+v"(v), "+v"(tk), "+v"(a0), "+v"(a1), "=&v"(pg) : "v"(cw) : "memory");
        if (!dry && !__all(min(min(min4(v), min4(a0)), min4(a1)) != 0u)) {
          Spin sp;
          unsigned quick = QUICK;
          while (ok) {
            if (f < NPUSH) __builtin_amdgcn_s_sleep(2);
            else if (quick) --quick;
            else slot_wait(9u);
            asm volatile("global_load_dwordx4 %0, %3, %4 sc0 sc1\n\tglobal_load_dwordx4 %1, %3, %5 sc1\n\tglobal_load_dwordx4 %2, %3, %5 offset:1024 sc1\n\ts_waitcnt vmcnt(0)"
                         : "=&v"(v), "=&v"(a0), "=&v"(a1) : "v"(vlin), "s"(src), "s"(fsrc) : "memory");
            if (__all(min(min(min4(v), min4(a0)), min4(a1)) != 0u)) break;
            if (sp.fail(tmo, seq)) ok = false;
          }
        }
        if (!dry) nn = __builtin_amdgcn_readfirstlane(tk) + NPW;
        if (!cleared && f >= 256u) wait_cleared();
        if (first && nn >= 1024u && !dry) {
          asm volatile("global_store_dwordx4 %0, %1, %2 sc0 sc1\n\ts_waitcnt vmcnt(0)" :: "v"(vidle), "v"(zero4), "s"(odst) : "memory");
          first = false;
        }
        f32x4 s0 = __builtin_bit_cast(f32x4, a0), s1 = __builtin_bit_cast(f32x4, a1);
        s0 += f32x4{__builtin_bit_cast(float, v[0] << 16), __builtin_bit_cast(float, v[0] & 0xffff0000u), __builtin_bit_cast(float, v[1] << 16), __builtin_bit_cast(float, v[1] & 0xffff0000u)};
        s1 += f32x4{__builtin_bit_cast(float, v[2] << 16), __builtin_bit_cast(float, v[2] & 0xffff0000u), __builtin_bit_cast(float, v[3] << 16), __builtin_bit_cast(float, v[3] & 0xffff0000u)};
        u32x4 o = ok ? nonzero(to_bf16x8(s0, s1)) : nanv;
        asm volatile("" : "+v"(o) : "s"(odst), "s"(ddst));
        if (dry) { dry = false; continue; }
        asm volatile("global_store_dwordx4 %0, %2, %3 sc0 sc1\n\tglobal_store_dwordx4 %1, %2, %4 sc1\n\ts_nop 1" :: "v"(vlin), "v"(vout), "v"(o), "s"(odst), "s"(ddst) : "memory");
        if (nn >= 1024u) break;
        if (first) { asm volatile("global_store_dwordx4 %0, %1, %2 sc0 sc1\n\ts_nop 1" :: "v"(vidle), "v"(zero4), "s"(odst) : "memory"); first = false; }
        asm volatile("global_store_dwordx4 %0, %1, %2 sc0 sc1\n\ts_nop 1" :: "v"(vidle), "v"(zero4), "s"(myws + OE_OFF + nn * 1024u) : "memory");
      }
      cleared = true;
    }
    unsigned lim = l == 1u ? LENP : LEN, lo = 0u;
    auto plan = [&]() {
      const bool on = lane >= 1 && lane < NR;
      const int rel = on ? (int)(pg - __builtin_amdgcn_readlane(pg, 1)) : 0;
      const int e = on ? (max(min(rel, 4000), -4000) * RATE >> 8) + (lane == 1 ? (int)(LENP - MARKP) + PBIAS : (int)(LEN - MARK)) : 0;
      int sum = 0;
#pragma unroll
      for (int i = 1; i < NR; ++i) sum += __builtin_amdgcn_readlane(e, i);
      const bool act = on && (lane == 1 || e * 7 < sum);
      const int ea = act ? e : 0;
      sum = 0;
#pragma unroll
      for (int i = 1; i < NR; ++i) sum += __builtin_amdgcn_readlane(ea, i);
      const int level = sum / (int)__popcll(__ballot(act));
      const unsigned h = act && lane != 1 ? (unsigned)max(level - e, 0) : 0u;
      unsigned H = 0u, hi = 0u;
#pragma unroll
      for (int i = 2; i < NR; ++i) {
        if ((unsigned)i == l) lo = H;
        H = min(H + __builtin_amdgcn_readlane(h, i), CAP);
        if ((unsigned)i == l) hi = H;
      }
      lim = l == 1u ? LENP - H : LEN + (hi - lo);
      planned = true;
    };
#pragma unroll 1
    for (;;) {
      if (!planned && nn >= (l == 1u ? LENP - CAP : LEN)) {
        Spin sp;
        while (ok && !__all(pg != 0u)) {
          __builtin_amdgcn_s_sleep(16);
          if (sp.fail(tmo, seq)) ok = false;
          CW_RELOAD();
        }
        if (ok) plan();
        planned = true;
      }
      if (nn >= lim) break;
      const unsigned n = nn;
      const char* src; const char* direct; char* dst;
      bool relay = false;
      if (l != 1u && n < NIN) {
        const unsigned o = ((0xEE50u >> ((n >> 6) * 2u)) & 3u) * 65536u + (n & 63u) * 1024u;
        const bool isw = (0xC6u >> (n >> 6)) & 1u;
        src = (isw ? wsrc : xsrc) + o; dst = xdst + (isw ? WS_OFF - XS_OFF : 0u) + o;
        direct = src;
      } else if (l != 1u && n >= LEN) {
        const unsigned f = (lo + (n - LEN)) & 1023u, o = out_off(rank ^ 1u, f);
        direct = partner_oe + f * 1024u;
        relay = trusted && ok;
        src = relay ? pout + o : direct; dst = outp + o;
      } else {
        const unsigned m = n - 1024u;
        const unsigned f = (l != 1u ? n - NIN : m < 1024u - CAP ? CAP + m : 1023u - m) & 1023u;
        src = peer + OE_OFF + f * 1024u; dst = outp + out_off(d, f);
        direct = src;
      }
      const bool isout = l == 1u || n >= NIN;
      const unsigned vsrc = relay ? vout : vlin, vdst = isout ? vout : vlin;
      const char* const cwp = (!cleared && isout) || (!planned && (l == 1u || n >= POLLN)) ? tk0 : tk0 + 128;
      asm volatile("global_load_dwordx4 %0, %[vs], %[src] sc0 sc1\n\tglobal_load_dword %2, %[co], %[cp] offset:1024 sc1\n\t" TICKET_ASM
                   : "=&v"(v), "=&v"(tk), "=&v"(cw)
                   : [vs] "v"(vsrc), [vl] "v"(vlin), [src] "s"(src), [co] "v"(coff), [cp] "s"(cwp), [one] "v"(one), [tk] "s"(tkp) : "memory");
      if (l != 1u && n == MARK) __hip_atomic_store(cwl + l, (unsigned)__builtin_amdgcn_s_memrealtime() | 1u, __ATOMIC_RELAXED, __HIP_MEMORY_SCOPE_AGENT);
      if (!planned && __all(pg != 0u)) plan();
      asm volatile("s_waitcnt vmcnt(0)\n\tv_mov_b32 %2, %3" : "+v"(v), "+v"(tk), "=&v"(pg) : "v"(cw) : "memory");
      if (!__all(min4(v) != 0u)) {
        Spin sp;
        while (ok) {
          if (isout || sp.it >= QUICK) slot_wait(9u);
          asm volatile("global_load_dwordx4 %0, %1, %2 sc0 sc1\n\ts_waitcnt vmcnt(0)" : "=&v"(v) : "v"(vlin), "s"(direct) : "memory");
          if (__all(min4(v) != 0u)) break;
          if (sp.fail(tmo, seq)) ok = false;
        }
      }
      nn = __builtin_amdgcn_readfirstlane(tk) + NPW;
      if (!cleared && isout) wait_cleared();
      if (!ok) v = nanv;
      asm volatile("global_store_dwordx4 %0, %1, %2 sc0 sc1\n\ts_nop 1" :: "v"(vdst), "v"(v), "s"(dst) : "memory");
      trusted = trusted || (ok && n >= NIN + 256u && n < LEN);
    }
  }
}

}

extern "C" void kernel_launch(int rank, int nranks, ncclComm_t comm,
                              hipStream_t stream,
                              void* const* d_in, const int* in_sizes, int n_in,
                              void* d_out, int out_size,
                              const int* in_dtypes, int out_dtype,
                              void* const* d_out_all,
                              void* const* d_ws_all, size_t ws_bytes) {
  if (nranks != NR || (unsigned)rank >= (unsigned)NR || n_in < 2 || ws_bytes < WS_USED ||
      (size_t)out_size * 2 != OUT_BYTES || (size_t)in_sizes[0] * 2 != IN_BYTES || (size_t)in_sizes[1] * 2 != IN_BYTES) {
    fprintf(stderr, "kernel_launch: unexpected problem shape\n");
    return;
  }
  Params P;
  const int lr = ((rank & 3) << 1) | (rank >> 2);
  for (int i = 0; i < NR; ++i) {
    const int lp = lr ^ i, pp = (lp >> 1) | ((lp & 1) << 2);
    P.ws[i] = (char*)d_ws_all[pp]; P.outs[i] = (char*)d_out_all[pp];
  }
  P.out = (char*)d_out;
  P.x = (const char*)d_in[0];
  P.w = (const char*)d_in[1];
  P.rank = lr;
  P.ws0 = P.ws[0];
  P.pad = 0;
  for (int i = 0; i < 6; ++i) P.pad2[i] = 0;
  gemm_ar_kernel<<<dim3(NWG), dim3(512), 0, stream>>>(P);
  HIP_CHECK(hipGetLastError());
}
```
